```python
import numpy as np

kernel_name = "gpu_gemm_rs_m8192_k8192_n8192_f32_gelu"


ABSMAX_RTOL = 0.001

COMPUTE_DTYPE = "float32"
EPILOGUE = "gelu"
IN_QUANT = None
OUT_QUANT = None
IN_DTYPES = {"x": "float32", "w_mat": "float32"}
WORLD_SIZE = 8

def setup_inputs():

    x = np.random.randn(8192, 8192).astype(np.float32)
    w_mat = (np.random.randn(8192, 8192) / np.sqrt(8192)).astype(np.float32)
    return {"x": x, "w_mat": w_mat}

def _epilogue(y):
    c = 0.7978845608028654
    return (0.5 * y * (1.0 + np.tanh(c * (y + 0.044715 * y ** 3)))).astype(np.float32)

def reference(x, w_mat):
    y = (x @ w_mat).astype(np.float32)
    return _epilogue(y)
```

```cpp
#include <hip/hip_runtime.h>
#include <rccl/rccl.h>
#include <cstdio>
#include <cstdint>
#include <type_traits>

#define HIP_CHECK(expr) do { hipError_t _e = (expr); if (_e != hipSuccess) { \
    fprintf(stderr, "HIP error %s at %s:%d\n", hipGetErrorString(_e), __FILE__, __LINE__); \
    return; } } while (0)

#define SCH_RATIO8 24
#define SCH_LEAD8 72
#define SCH_XO8 160
#define SCH_XOR8 40

namespace {

typedef _Float16 h8 __attribute__((ext_vector_type(8)));
typedef float f4 __attribute__((ext_vector_type(4)));
typedef unsigned u4 __attribute__((ext_vector_type(4)));
#define LAS __attribute__((address_space(3)))

constexpr int NR = 8;
constexpr int NDIM = 8192;
constexpr int KSH = 1024;
constexpr int NREL = 2;
#define NPUSH_WG 24
#define PUSH_LIM 14
constexpr int NPUSH = NPUSH_WG;
static_assert(NPUSH % 8 == 0 && NPUSH <= 32, "pusher count");
constexpr int NCH_IN = 128 + 256 + NREL * 32;
constexpr int NCH_X = 128 + (8 - NREL) * 32 + NREL * 8;
constexpr int GRID = 256;
constexpr int NGW = GRID - NPUSH;
constexpr int NCONV = 768;
constexpr unsigned long long DEADLINE_TICKS = 20000000ull;

constexpr unsigned OFF_TICKET = 0, OFF_TMO = 128, OFF_TQ = 256;
constexpr unsigned OFF_LKT = 1024;
constexpr unsigned OFF_LXF = 4096;
constexpr unsigned OFF_LWF = 8192;
constexpr unsigned OFF_DONE = 16384;
constexpr unsigned OFF_XF = 65536;
constexpr unsigned OFF_WF = 65536 + 4096;
constexpr unsigned OFF_DBG = 262144;
constexpr unsigned MIB = 1u << 20;
constexpr unsigned OFF_XL = 1 * MIB;
constexpr unsigned OFF_WL = 17 * MIB;
constexpr unsigned OFF_XR = 33 * MIB;
constexpr unsigned OFF_WR = 65 * MIB;
constexpr unsigned PAR_STRIDE = 96 * MIB;
constexpr unsigned WS_USED = 33 * MIB + 2 * PAR_STRIDE;

constexpr int STAGE_B = 49152;
constexpr int L_CTL = 3 * STAGE_B;
constexpr int L_Q = L_CTL;
constexpr int L_PF = L_CTL + 1024;
constexpr int L_RED = L_CTL + 1792;
constexpr int L_BC = L_CTL + 1856;
constexpr int L_META = L_CTL + 1920;
constexpr int LDS_BYTES = L_CTL + 2048;

#define GLDS_AUX 16

struct ConvTab { unsigned short v[NCONV]; };
constexpr int conv_key(int id) {
    const int b = id / 96, r = id % 96;
    int k = r < 32 ? b * SCH_RATIO8 - 4 : ((r - 32) >> 1) * 8 + b * SCH_RATIO8 - (b >= 8 - NREL ? SCH_LEAD8 : 0);
    k += 64;
    return k < 0 ? 0 : (k > 2047 ? 2047 : k);
}
constexpr ConvTab make_conv_tab() {
    ConvTab t{}; int cnt[2049] = {};
    for (int i = 0; i < NCONV; ++i) cnt[conv_key(i) + 1]++;
    for (int k = 0; k < 2048; ++k) cnt[k + 1] += cnt[k];
    for (int i = 0; i < NCONV; ++i) t.v[cnt[conv_key(i)]++] = (unsigned short)i;
    return t;
}
__constant__ const ConvTab conv_tab = make_conv_tab();

struct Params {
    const float* x; const float* w;
    void* const* ws_all; void* const* out_all;
    int rank;
    int ratio8, lead8, xo8, xor8;
};

#define GAS __attribute__((address_space(1)))
__device__ __forceinline__ unsigned ld_sys(const unsigned* p) { return __hip_atomic_load((const GAS unsigned*)p, __ATOMIC_RELAXED, __HIP_MEMORY_SCOPE_SYSTEM); }
__device__ __forceinline__ unsigned ld_agent(const unsigned* p) { return __hip_atomic_load((const GAS unsigned*)p, __ATOMIC_RELAXED, __HIP_MEMORY_SCOPE_AGENT); }
__device__ __forceinline__ void st_sys(unsigned* p, unsigned v) { __hip_atomic_store((GAS unsigned*)p, v, __ATOMIC_RELAXED, __HIP_MEMORY_SCOPE_SYSTEM); }
__device__ __forceinline__ void st_agent(unsigned* p, unsigned v) { __hip_atomic_store((GAS unsigned*)p, v, __ATOMIC_RELAXED, __HIP_MEMORY_SCOPE_AGENT); }
__device__ __forceinline__ unsigned add_agent(unsigned* p) { return __hip_atomic_fetch_add((GAS unsigned*)p, 1u, __ATOMIC_RELAXED, __HIP_MEMORY_SCOPE_AGENT); }
__device__ __forceinline__ int rfl(int v) { return __builtin_amdgcn_readfirstlane(v); }
__device__ __forceinline__ unsigned rflu(unsigned v) { return (unsigned)__builtin_amdgcn_readfirstlane((int)v); }
__device__ __forceinline__ char* rfl_ptr(char* p) {
    const unsigned long long v = (unsigned long long)p;
    return (char*)(((unsigned long long)rflu((unsigned)(v >> 32)) << 32) | rflu((unsigned)v));
}
#define WAIT_V(n) asm volatile("s_waitcnt vmcnt(" #n ")" ::: "memory")
#define WAIT_L0() asm volatile("s_waitcnt lgkmcnt(0)" ::: "memory")
#define BAR() do { asm volatile("" ::: "memory"); __builtin_amdgcn_s_barrier(); asm volatile("" ::: "memory"); } while (0)

#define STAMP(ws, slot) do { } while (0)

__device__ __forceinline__ int rb32_of(int t, int rbl) {
    return rbl < 4 ? 4 * t + rbl : 4 * ((((t >> 2) ^ 1) << 2) + (rbl - 4)) + (t & 3);
}

__device__ __forceinline__ int block_scale(float am, LAS unsigned char* lds, int par, int& e) {
#pragma unroll
    for (int o = 32; o >= 1; o >>= 1) am = fmaxf(am, __shfl_xor(am, o));
    LAS float* red = (LAS float*)(lds + L_RED) + par * 8;
    if ((threadIdx.x & 63) == 0) red[threadIdx.x >> 6] = am;
    __syncthreads();
    float m = red[0];
#pragma unroll
    for (int i = 1; i < 8; ++i) m = fmaxf(m, red[i]);
    const int ex = __builtin_amdgcn_frexp_expf(m);
    e = rfl((15 - ex) & ~3);
    return rfl(m == 0.0f ? 0 : e + 257);
}
__device__ __forceinline__ float amax4(f4 v, float a) {
    return fmaxf(fmaxf(fmaxf(fabsf(v[0]), fabsf(v[1])), fmaxf(fabsf(v[2]), fabsf(v[3]))), a);
}
__device__ __forceinline__ _Float16 cvt1(float v, int e) { return (_Float16)__builtin_amdgcn_ldexpf(v, e); }

__device__ __forceinline__ float gelu_tanh(float y) {
    const float u = 0.7978845608028654f * (y + 0.044715f * y * y * y);
    const float ex = __expf(-2.0f * u);
    return y * __builtin_amdgcn_rcpf(1.0f + ex);
}

__global__ void __launch_bounds__(512, 2) fused_gemm_gelu(Params P) {
    extern __shared__ __attribute__((aligned(16))) unsigned char smem[];
    LAS unsigned char* lds = (LAS unsigned char*)smem;
    const int tid = threadIdx.x, lane = tid & 63, wid = rfl(tid >> 6);
    const int me = P.rank;
    char* ws = rfl_ptr((char*)P.ws_all[me]);
    unsigned* tmo = (unsigned*)(ws + OFF_TMO);

    if (tid == 0) {
        const unsigned t = add_agent((unsigned*)(ws + OFF_TICKET));
        *(LAS unsigned*)(lds + L_BC) = t / GRID + 1u;
    }
    __syncthreads();
    const unsigned call = rflu(*(LAS unsigned*)(lds + L_BC));
    const unsigned seq = call & 0x3FFFFFu;
    const unsigned par = call & 1u;
    const unsigned xf_off = OFF_XF + par * 65536u, wf_off = OFF_WF + par * 65536u;
    const unsigned xr_off = OFF_XR + par * PAR_STRIDE, wr_off = OFF_WR + par * PAR_STRIDE;
    const int bi = blockIdx.x;
    STAMP(ws, 0);

    if (bi < NPUSH) {
        auto chunk_key = [&](int id, bool ing) {
            if (id < 128) { const int b = id >> 4, rbl = (id >> 1) & 7; return rbl >= 4 ? b * P.ratio8 - 4 : P.xo8 + b * P.xor8; }
            const int nW = ing ? 256 : (8 - NREL) * 32, r = id - 128;
            if (r < nW) return (r & 31) * 8 + (r >> 5) * P.ratio8;
            const int q = r - nW;
            if (ing) return (q & 7) * 32 + 12 + (8 - NREL + (q >> 5)) * P.ratio8;
            return (q & 7) * 32 + 12 + (8 - NREL + (q >> 3)) * P.ratio8 - P.lead8;
        };
        typedef int i4 __attribute__((ext_vector_type(4)));
        LAS int* keys = (LAS int*)lds;
        LAS unsigned short* order = (LAS unsigned short*)(lds + 4096);
        const int k0 = tid < NCH_IN ? (chunk_key(tid, true) + 4096) * 512 + tid : 0x7fffffff;
        const int k1 = tid < NCH_X ? (chunk_key(tid, false) + 4096) * 512 + tid : 0x7fffffff;
        keys[tid] = k0; keys[512 + tid] = k1;
        __syncthreads();
        int r0 = 0, r1 = 0;
#pragma unroll 8
        for (int j = 0; j < NCH_IN / 4; ++j) {
            const i4 a = ((LAS i4*)keys)[j], c = ((LAS i4*)keys)[128 + j];
            r0 += (a.x < k0 ? 1 : 0) + (a.y < k0 ? 1 : 0) + (a.z < k0 ? 1 : 0) + (a.w < k0 ? 1 : 0);
            r1 += (c.x < k1 ? 1 : 0) + (c.y < k1 ? 1 : 0) + (c.z < k1 ? 1 : 0) + (c.w < k1 ? 1 : 0);
        }
        if (tid < NCH_IN) order[r0] = (unsigned short)tid;
        if (tid < NCH_X) order[512 + r1] = (unsigned short)tid;
        __syncthreads();
        bool dead = false;
        const unsigned loff = (unsigned)lane * 16u;
        u4 buf[4][4];
#pragma unroll
        for (int j = 0; j < 4; ++j)
#pragma unroll
            for (int i = 0; i < 4; ++i) buf[j][i] = (u4){0u, 0u, 0u, 0u};
        const int v24 = (bi * 4 + wid) % 24, l0 = v24 < 21 ? v24 % 7 : v24 - 21;
        for (int lk = 0; lk < 7 && wid < 4 && !dead; ++lk) {
        const int lx = (l0 + lk) % 7;
        const int t = lx < 3 ? (me & 4) + ((me + 1 + lx) & 3) : ((me & 4) ^ 4) + ((me + lx) & 3);
        const bool ingrp = lx < 3;
        const int nch = ingrp ? NCH_IN : NCH_X;
        unsigned* tkp = (unsigned*)(ws + OFF_LKT + lx * 128);
        const unsigned tkbase = (call - 1u) * (unsigned)(nch + 4 * NPUSH);
        char* wsd = rfl_ptr((char*)P.ws_all[t & 7]);
        const LAS unsigned short* ord = order + (ingrp ? 0 : 512);
        struct Item { unsigned src, dst, wfl, dfl; };
        auto decode = [&](int tk) {
            const int id = rfl((int)ord[tk]);
            Item it;
            if (id < 128) {
                const int b = id >> 4, rbl = (id >> 1) & 7, ks = id & 1, rb = rb32_of(t, rbl);
                it.src = OFF_XL + (unsigned)(rb * 8 + b) * 65536u + ks * 32768u; it.wfl = OFF_LXF + (unsigned)(rb * 8 + b) * 4u;
                it.dst = xr_off + (unsigned)((rbl * 8 + me) * 8 + b) * 65536u + ks * 32768u;
                it.dfl = xf_off + (unsigned)((rbl * 2 + ks) * 64 + me * 8 + b) * 4u;
            } else {
                const int nW = ingrp ? 256 : (8 - NREL) * 32, r = id - 128, q = r - nW;
                if (r < nW || !ingrp) {
                    int b = r >> 5, g = r & 31;
                    if (r >= nW) { b = 8 - NREL + (q >> 3); g = 4 * (q & 7) + (((t & 3) - b) & 3); }
                    const int cg = (t >> 2) * 32 + g;
                    it.src = OFF_WL + (unsigned)(cg * 8 + b) * 32768u; it.wfl = OFF_LWF + (unsigned)(cg * 8 + b) * 4u;
                    it.dst = wr_off + (unsigned)((g * 8 + me) * 8 + b) * 32768u; it.dfl = wf_off + (unsigned)(g * 64 + me * 8 + b) * 4u;
                } else {
                    const int b = 8 - NREL + (q >> 5), ur = (((me >> 2) ^ 1) << 2) + ((q >> 3) & 3), g = 4 * (q & 7) + (((me & 3) - b) & 3);
                    it.src = it.dst = wr_off + (unsigned)((g * 8 + ur) * 8 + b) * 32768u;
                    it.wfl = it.dfl = wf_off + (unsigned)(g * 64 + ur * 8 + b) * 4u;
                }
            }
            return it;
        };
        auto draw = [&]() {
            unsigned v = 0;
            if (lane == 0) v = add_agent(tkp);
            return v;
        };
        int tk = (int)(rflu(draw()) - tkbase);
        Item cur{0u, 0u, 0u, 0u}, prv{0u, 0u, 0u, 0u}; unsigned cur_val = 0u, prv_val = 0u; bool cur_ok = false, prv_ok = false;
        unsigned fv = 0u;
        bool have = tk < nch;
        Item nxt{0u, 0u, 0u, 0u};
        if (have) nxt = decode(tk);
        for (;;) {
            cur_ok = false;
            if (have) {
                if (!prv_ok) {
                    const unsigned long long t0 = __builtin_amdgcn_s_memrealtime();
                    for (;;) {
                        fv = rflu(ld_sys((const unsigned*)(ws + nxt.wfl)));
                        if ((fv >> 10) == seq) break;
                        if (rflu(ld_agent(tmo)) == call || __builtin_amdgcn_s_memrealtime() - t0 > DEADLINE_TICKS) {
                            if (lane == 0) st_agent(tmo, call);
                            dead = true; break;
                        }
                        __builtin_amdgcn_s_sleep(2);
                    }
                    if (dead) break;
                }
                if ((fv >> 10) == seq) { cur = nxt; cur_val = fv; cur_ok = true; }
            } else if (!prv_ok) break;
            const __amdgpu_buffer_rsrc_t rl = __builtin_amdgcn_make_buffer_rsrc(ws, 0, cur_ok ? (int)WS_USED : 0, 0x00020000);
            const __amdgpu_buffer_rsrc_t rsp = __builtin_amdgcn_make_buffer_rsrc(wsd, 0, prv_ok ? (int)WS_USED : 0, 0x00020000);
            const __amdgpu_buffer_rsrc_t rsc = __builtin_amdgcn_make_buffer_rsrc(wsd, 0, cur_ok ? (int)WS_USED : 0, 0x00020000);
            unsigned tkv = 0u, fvv = 0u;
            if (cur_ok && lane == 0) {
                const unsigned one = 1u;
                asm volatile("global_atomic_add %0, %1, %2, off sc0" : "=v"(tkv) : "v"(tkp), "v"(one) : "memory");
            }
#pragma unroll
            for (int st = 0; st < 8; ++st) {
                const int j = st & 3;
                if (st == 4) {
                    asm volatile("s_waitcnt vmcnt(32)" : "+v"(tkv) :: "memory");
                    if (cur_ok) { tk = (int)(rflu(tkv) - tkbase); have = tk < nch; if (have) nxt = decode(tk); }
                    fvv = ld_sys((const unsigned*)(ws + nxt.wfl));
                }
                if (st == 6) {
                    WAIT_V(18);
                    if (prv_ok && lane == 0) st_sys((unsigned*)(wsd + prv.dfl), prv_val);
                }
#pragma unroll
                for (int i = 0; i < 4; ++i) {
                    asm volatile("s_waitcnt vmcnt(%0)" :: "n"(PUSH_LIM) : "memory");
                    if (st < 4) __builtin_amdgcn_raw_buffer_store_b128(buf[j][i], rsp, (int)(prv.dst + loff + (unsigned)((st + 4) * 4096 + i * 1024)), 0, 17);
                    else __builtin_amdgcn_raw_buffer_store_b128(buf[j][i], rsc, (int)(cur.dst + loff + (unsigned)((st - 4) * 4096 + i * 1024)), 0, 17);
                    buf[j][i] = __builtin_amdgcn_raw_buffer_load_b128(rl, (int)(cur.src + loff + (unsigned)(st * 4096 + i * 1024)), 0, 16);
                }
            }
            fv = rflu(fvv);
            prv = cur; prv_val = cur_val; prv_ok = cur_ok;
        }
        }
        if (bi == 0 && wid == 4) {
            const unsigned* dp = (const unsigned*)(ws + OFF_DONE);
            const unsigned long long t0 = __builtin_amdgcn_s_memrealtime();
            bool bad = dead;
            while (!bad) {
                const unsigned a = ld_sys(dp + lane), b = ld_sys(dp + 64 + lane);
                if (__all((int)(a - call) >= 0 && (int)(b - call) >= 0)) break;
                if (rflu(ld_agent(tmo)) == call || __builtin_amdgcn_s_memrealtime() - t0 > DEADLINE_TICKS) {
                    if (lane == 0) st_agent(tmo, call);
                    bad = true; break;
                }
                __builtin_amdgcn_s_sleep(2);
            }
            __builtin_amdgcn_fence(__ATOMIC_ACQUIRE, "");
            if (bad && lane == 0) {
                float* o = (float*)P.out_all[me];
                o[1023 * (size_t)NDIM + (((me >> 2) ^ 1) * 4096 + 4095)] = __builtin_nanf("");
            }
        }
        return;
    }

    {
        const __amdgpu_buffer_rsrc_t rsl = __builtin_amdgcn_make_buffer_rsrc(ws, 0, (int)WS_USED, 0x00020000);
        int rp = 0;
        for (int it = bi - NPUSH; it < NCONV; it += NGW, rp ^= 1) {
            const int id = conv_tab.v[it], b = id / 96, r = id % 96;
            int e, code; unsigned fo;
            if (r < 32) {
                const int rb = r;
                f4 v[16]; float am = 0.f;
#pragma unroll
                for (int i = 0; i < 8; ++i) {
                    const int q = i * 512 + tid, ks = q >> 11, row = (q >> 3) & 255, c = (q & 7) ^ (row & 7);
                    const float* s = P.x + (size_t)(rb * 256 + row) * KSH + b * 128 + ks * 64 + c * 8;
                    v[2 * i] = *(const f4*)s; v[2 * i + 1] = *(const f4*)(s + 4);
                }
#pragma unroll
                for (int i = 0; i < 16; ++i) am = amax4(v[i], am);
                code = block_scale(am, lds, rp, e);
                const unsigned base = OFF_XL + (unsigned)(rb * 8 + b) * 65536u + (unsigned)tid * 16u;
#pragma unroll
                for (int i = 0; i < 8; ++i) {
                    h8 o;
#pragma unroll
                    for (int j = 0; j < 4; ++j) { o[j] = cvt1(v[2 * i][j], e); o[4 + j] = cvt1(v[2 * i + 1][j], e); }
                    __builtin_amdgcn_raw_buffer_store_b128(__builtin_bit_cast(u4, o), rsl, (int)(base + i * 8192u), 0, 17);
                }
                fo = OFF_LXF + (unsigned)(rb * 8 + b) * 4u;
            } else {
                const int g2 = r - 32, cg = (g2 >> 1) + (g2 & 1) * 32;
                const int n4 = tid & 31, kc = tid >> 5;
                f4 v[8]; float am = 0.f;
#pragma unroll
                for (int j = 0; j < 8; ++j) v[j] = *(const f4*)(P.w + (size_t)(b * 128 + kc * 8 + j) * NDIM + cg * 128 + n4 * 4);
#pragma unroll
                for (int j = 0; j < 8; ++j) am = amax4(v[j], am);
                code = block_scale(am, lds, rp, e);
#pragma unroll
                for (int e4 = 0; e4 < 4; ++e4) {
                    const int n = n4 * 4 + e4; h8 o;
#pragma unroll
                    for (int j = 0; j < 8; ++j) o[j] = cvt1(v[j][e4], e);
                    *(LAS h8*)(lds + (kc >> 3) * 16384 + n * 128 + (((kc & 7) ^ (n & 7)) << 4)) = o;
                }
                __syncthreads();
                const unsigned base = OFF_WL + (unsigned)(cg * 8 + b) * 32768u + (unsigned)tid * 16u;
#pragma unroll
                for (int i = 0; i < 4; ++i) {
                    const u4 d = *(LAS u4*)(lds + i * 8192 + tid * 16);
                    __builtin_amdgcn_raw_buffer_store_b128(d, rsl, (int)(base + i * 8192u), 0, 17);
                }
                fo = OFF_LWF + (unsigned)(cg * 8 + b) * 4u;
            }
            WAIT_V(0);
            __syncthreads();
            if (tid == 0) st_sys((unsigned*)(ws + fo), (seq << 10) | (unsigned)code);
        }
    }
    STAMP(ws, 1);

    const int wm = wid >> 1, wn = wid & 1, fr = lane & 15, fq = lane >> 4;
    const int fo0 = fr * 128 + ((fq ^ (fr & 7)) << 4), fo1 = fo0 ^ 64;
    const int abase = wm * 8192, bbase = 32768 + wn * 8192;
    constexpr unsigned TQ_PER_CALL = 256u + (32u - NPUSH);
    int xcd = bi & 7;
    int u = (bi >> 3) - ((NPUSH - xcd + 7) >> 3);
    int ntile_done = 0;
    bool pool = u >= 28;
    for (;;) {
        if (pool) {
            if (tid == 0) {
                const unsigned o = add_agent((unsigned*)(ws + OFF_TQ));
                *(LAS unsigned*)(lds + L_BC + 4) = o - (call - 1u) * TQ_PER_CALL;
            }
            __syncthreads();
            const unsigned nx = rflu(*(LAS unsigned*)(lds + L_BC + 4));
            if (nx >= 32u) break;
            xcd = (int)(nx >> 2); u = 28 + (int)(nx & 3u);
        }
        pool = true;
        const int g = 4 * xcd + ((u & 15) >> 2);
        const int rbl = (u < 16 ? 4 : 0) + (u & 3);
        const int rb = rb32_of(me, rbl), cg = (me >> 2) * 32 + g;
        f4 acc[4][4];
#pragma unroll
        for (int m = 0; m < 4; ++m)
#pragma unroll
            for (int n = 0; n < 4; ++n) acc[m][n] = (f4){0.f, 0.f, 0.f, 0.f};
        const int ls = lane >> 3, lb = lane & 7;
        const unsigned* xfp = (const unsigned*)(ws + (ls == me ? OFF_LXF + (unsigned)(rb * 8 + lb) * 4u : xf_off + (unsigned)(rbl * 128 + lane) * 4u));
        const unsigned* xgp = (const unsigned*)(ws + (ls == me ? OFF_LXF + (unsigned)(rb * 8 + lb) * 4u : xf_off + (unsigned)(rbl * 128 + 64 + lane) * 4u));
        const unsigned* wfp = (const unsigned*)(ws + (ls == me ? OFF_LWF + (unsigned)(cg * 8 + lb) * 4u : wf_off + (unsigned)(g * 64 + lane) * 4u));
        const unsigned blk_a = ls == me ? OFF_XL + (unsigned)(rb * 8 + lb) * 65536u : xr_off + (unsigned)(rbl * 64 + lane) * 65536u;
        const unsigned blk_b = ls == me ? OFF_WL + (unsigned)(cg * 8 + lb) * 32768u : wr_off + (unsigned)(g * 64 + lane) * 32768u;
        unsigned long long seen = 0ull; int nq = 0, Ecur = 0, Emin = 0, haveE = 0, bad = 0;
        int polled = 0;
        int cs = 0, is = 0, avail = 0, ep = 0, allk = 0; unsigned qa = 0u, qb = 0u, meta = 0u, metav = 0u;
        int rd = 0, gst = 0;

        auto absorb = [&](unsigned xv, unsigned xw, unsigned wv) {
            unsigned long long rdy = __ballot((xv >> 10) == seq && (xw >> 10) == seq && (wv >> 10) == seq) & ~seen;
            seen |= rdy;
            if (rdy == 0ull) return;
            LAS unsigned* q = (LAS unsigned*)(lds + L_Q);
            const int cx = (int)(xv & 1023u), cw = (int)(wv & 1023u), el = cx + cw - 514;
            if (!haveE) {
                const unsigned long long nz = __ballot(cx != 0 && cw != 0) & rdy;
                if (nz) { haveE = 1; Ecur = __builtin_amdgcn_readlane(el, __builtin_ctzll(nz)); Emin = Ecur; }
            }
            const unsigned long long same = __ballot(cx != 0 && cw != 0 && el == Ecur) & rdy;
            if ((same >> lane) & 1ull) {
                const int pos = nq + __builtin_popcountll(same & ((1ull << lane) - 1ull));
                q[pos] = blk_a; q[64 + pos] = blk_b;
            }
            nq += __builtin_popcountll(same);
            rdy &= ~same;
            while (rdy) {
                const int l = __builtin_ctzll(rdy); rdy &= rdy - 1;
                if (__builtin_amdgcn_readlane(cx, l) == 0 || __builtin_amdgcn_readlane(cw, l) == 0) continue;
                const int e = __builtin_amdgcn_readlane(el, l);
                if (e > Emin + 60) continue;
                const int delta = e - Ecur; Ecur = e; Emin = min(Emin, e);
                if (lane == l) { q[nq] = blk_a | ((unsigned)delta & 0xFFFFu); q[64 + nq] = blk_b; }
                ++nq;
            }
        };
        auto publish = [&]() {
            const bool all = (seen == ~0ull) || bad;
            ((LAS unsigned*)(lds + L_META))[ep & 1] = (unsigned)nq | (all ? 128u : 0u) | (bad ? 256u : 0u) | ((unsigned)(Ecur + 1024) << 16);
        };
        auto fetch_req = [&]() {
            LAS unsigned* q = (LAS unsigned*)(lds + L_Q);
            qa = q[lane]; qb = q[64 + lane];
            metav = ((LAS unsigned*)(lds + L_META))[ep & 1];
            ++ep;
        };
        auto fetch_use = [&]() { meta = rflu(metav); avail = 2 * (int)(meta & 127u); allk = (int)((meta >> 7) & 1u); };
        const char* ga = nullptr; const char* gb = nullptr; int gdst = 0;
        const char* wsa = ws + wid * 4096 + lane * 16; const char* wsb = ws + wid * 2048 + lane * 16;
        auto issue_addr = [&]() {
            const unsigned ks = (unsigned)is & 1u;
            ga = wsa + (((unsigned)__builtin_amdgcn_readlane((int)qa, is >> 1) & 0xFFFF0000u) + ks * 32768u);
            gb = wsb + ((unsigned)__builtin_amdgcn_readlane((int)qb, is >> 1) + ks * 16384u);
            gdst = gst; gst = gst == 2 * STAGE_B ? 0 : gst + STAGE_B;
            ++is;
        };
#define GLDS(gp, lp, off) __builtin_amdgcn_global_load_lds((const unsigned*)(gp), (LAS unsigned*)(lp), 16, off, GLDS_AUX)
        auto issue_a = [&](int h) {
            if (h == 0) { GLDS(ga, lds + gdst + wid * 4096, 0); GLDS(ga, lds + gdst + wid * 4096, 1024); }
            else { GLDS(ga, lds + gdst + wid * 4096, 2048); GLDS(ga, lds + gdst + wid * 4096, 3072); }
        };
        auto issue_b = [&](int) { GLDS(gb, lds + gdst + 32768 + wid * 2048, 0); GLDS(gb, lds + gdst + 32768 + wid * 2048, 1024); };
        auto issue = [&]() { issue_addr(); issue_a(0); issue_a(1); issue_b(0); };
        h8 a0[4], b0[4], a1[4], b1[4];
        auto read_frags = [&](h8 (&af)[4], h8 (&bf)[4], int st, int fo) {
            LAS unsigned char* sp = lds + st + fo;
#pragma unroll
            for (int n = 0; n < 4; ++n) bf[n] = *(LAS h8*)(sp + bbase + n * 2048);
#pragma unroll
            for (int m = 0; m < 4; ++m) af[m] = *(LAS h8*)(sp + abase + m * 2048);
        };
#define MMA_ROW(m, af, bf) do { _Pragma("unroll") for (int n = 0; n < 4; ++n) \
            asm volatile("v_mfma_f32_16x16x32_f16 %0, %1, %2, %0" : "+v"(acc[m][n]) : "v"(bf[n]), "v"(af[m])); } while (0)
#define SCHED() __builtin_amdgcn_sched_barrier(0)
#define ACC_FENCE() do { SCHED(); asm volatile("s_nop 15\n\ts_nop 15" : "+v"(acc[0][0]), "+v"(acc[0][1]), "+v"(acc[0][2]), "+v"(acc[0][3]), \
            "+v"(acc[1][0]), "+v"(acc[1][1]), "+v"(acc[1][2]), "+v"(acc[1][3]), "+v"(acc[2][0]), "+v"(acc[2][1]), "+v"(acc[2][2]), "+v"(acc[2][3]), \
            "+v"(acc[3][0]), "+v"(acc[3][1]), "+v"(acc[3][2]), "+v"(acc[3][3])); SCHED(); } while (0)
        int pre = 0;
        auto step = [&]() {
            __builtin_amdgcn_s_waitcnt(0xC07F);
            MMA_ROW(0, a0, b0);
            SCHED();
            read_frags(a1, b1, rd, fo1);
            SCHED();
            MMA_ROW(1, a0, b0);
            SCHED();
            const int upd = allk ^ 1;
            if (upd && wid == 0) {
                if (polled == 2) { absorb(((LAS unsigned*)(lds + L_PF))[lane], ((LAS unsigned*)(lds + L_PF))[128 + lane], ((LAS unsigned*)(lds + L_PF))[64 + lane]); polled = 0; }
                publish();
            }
            SCHED();
            MMA_ROW(2, a0, b0);
            SCHED();
            pre = is > cs + 1 ? 1 : 0;
            const bool deep = is > cs + 2;
            const bool go = is < avail;
            if (go) issue_addr();
            rd = rd == 2 * STAGE_B ? 0 : rd + STAGE_B;
            SCHED();
            MMA_ROW(3, a0, b0);
            SCHED();
            if (deep) WAIT_V(6); else WAIT_V(0);
            __builtin_amdgcn_s_waitcnt(0xC07F);
            BAR();
            MMA_ROW(0, a1, b1);
            SCHED();
            if (upd) fetch_req();
            if (pre) read_frags(a0, b0, rd, fo0);
            SCHED();
            MMA_ROW(1, a1, b1);
            SCHED();
            if (upd && wid == 0) {
                if (polled == 1) polled = 2;
                else if (seen != ~0ull) {
                    __builtin_amdgcn_global_load_lds(xfp, (LAS unsigned*)(lds + L_PF), 4, 0, 17);
                    __builtin_amdgcn_global_load_lds(wfp, (LAS unsigned*)(lds + L_PF + 256), 4, 0, 17);
                    __builtin_amdgcn_global_load_lds(xgp, (LAS unsigned*)(lds + L_PF + 512), 4, 0, 17);
                    polled = 1;
                }
            }
            if (go) issue_a(0);
            SCHED();
            MMA_ROW(2, a1, b1);
            SCHED();
            if (go) issue_a(1);
            SCHED();
            MMA_ROW(3, a1, b1);
            SCHED();
            if (go) issue_b(0);
            if (upd) {
                fetch_use();
                while (is < avail && is <= cs + 3) issue();
            }
            ++cs;
        };
        auto fast_step = [&](auto first) {
            constexpr bool FIRST = decltype(first)::value;
            __builtin_amdgcn_s_waitcnt(0xC07F);
            MMA_ROW(0, a0, b0);
            SCHED();
            read_frags(a1, b1, rd, fo1);
            SCHED();
            MMA_ROW(1, a0, b0);
            SCHED();
            const int upd = allk ^ 1;
            if (FIRST && upd && wid == 0) {
                if (polled == 2) { absorb(((LAS unsigned*)(lds + L_PF))[lane], ((LAS unsigned*)(lds + L_PF))[128 + lane], ((LAS unsigned*)(lds + L_PF))[64 + lane]); polled = 0; }
                publish();
            }
            issue_addr();
            SCHED();
            MMA_ROW(2, a0, b0);
            rd = rd == 2 * STAGE_B ? 0 : rd + STAGE_B;
            MMA_ROW(3, a0, b0);
            SCHED();
            WAIT_V(6);
            __builtin_amdgcn_s_waitcnt(0xC07F);
            BAR();
            MMA_ROW(0, a1, b1);
            SCHED();
            read_frags(a0, b0, rd, fo0);
            if (FIRST && upd) {
                fetch_req();
                if (wid == 0) {
                    if (polled == 1) polled = 2;
                    else if (seen != ~0ull) {
                        __builtin_amdgcn_global_load_lds(xfp, (LAS unsigned*)(lds + L_PF), 4, 0, 17);
                        __builtin_amdgcn_global_load_lds(wfp, (LAS unsigned*)(lds + L_PF + 256), 4, 0, 17);
                        __builtin_amdgcn_global_load_lds(xgp, (LAS unsigned*)(lds + L_PF + 512), 4, 0, 17);
                        polled = 1;
                    }
                }
            }
            if (!FIRST) polled = polled == 1 ? 2 : polled;
            SCHED();
            MMA_ROW(1, a1, b1);
            SCHED();
            issue_a(0);
            SCHED();
            MMA_ROW(2, a1, b1);
            SCHED();
            issue_a(1);
            SCHED();
            MMA_ROW(3, a1, b1);
            SCHED();
            issue_b(0);
            if (FIRST && upd) fetch_use();
            ++cs;
        };
        int delta = 0;
        auto restart = [&]() {
            while (cs == is && !(allk && cs == avail)) {
                if (wid == 0) {
                    const int nq0 = nq;
                    WAIT_V(0); polled = 0;
                    const unsigned long long t0 = __builtin_amdgcn_s_memrealtime();
                    for (;;) {
                        const unsigned xv = ld_sys(xfp), xw = ld_sys(xgp), wv = ld_sys(wfp);
                        absorb(xv, xw, wv);
                        if (nq > nq0 || seen == ~0ull) break;
                        if (rflu(ld_agent(tmo)) == call || __builtin_amdgcn_s_memrealtime() - t0 > DEADLINE_TICKS) {
                            if (lane == 0) st_agent(tmo, call);
                            bad = 1; break;
                        }
                        __builtin_amdgcn_s_sleep(2);
                    }
                    publish();
                    WAIT_L0();
                }
                BAR();
                fetch_req(); fetch_use();
                while (is < avail && is <= cs + 2) issue();
            }
            pre = cs == is ? 2 : 1;
            if (pre == 1) {
                if (is > cs + 2) WAIT_V(12); else if (is > cs + 1) WAIT_V(6); else WAIT_V(0);
                BAR();
                read_frags(a0, b0, rd, fo0);
            }
        };
        auto prepare = [&]() {
            if (!pre) restart();
            delta = pre == 2 ? 0 : (int)(short)__builtin_amdgcn_readlane((int)qa, cs >> 1);
        };
        prepare();
        while (pre != 2) {
            const int dc = max(delta, -250), d1 = dc >> 1, d2 = dc - d1;
            const float s1 = __builtin_bit_cast(float, (unsigned)(127 + d1) << 23), s2 = __builtin_bit_cast(float, (unsigned)(127 + d2) << 23);
            ACC_FENCE();
#pragma unroll
            for (int m = 0; m < 4; ++m)
#pragma unroll
                for (int n = 0; n < 4; ++n) acc[m][n] = (acc[m][n] * s1) * s2;
            ACC_FENCE();
            do {
                delta = 0;
                while (is == cs + 3 && avail >= cs + 5 && delta == 0) {
                    fast_step(std::true_type{});
                    fast_step(std::false_type{});
                    delta = (int)(short)__builtin_amdgcn_readlane((int)qa, cs >> 1);
                }
                if (delta == 0) {
                    step();
                    if (!pre) restart();
                    step();
                    prepare();
                }
            } while (pre != 2 && delta == 0);
        }
        ACC_FENCE();
        STAMP(ws, 2 + 2 * ntile_done);
        {
            const int E = (int)(meta >> 16) - 1024;
            const bool isbad = (meta & 256u) != 0u || rflu(ld_agent(tmo)) == call;
            const int owner = rbl < 4 ? me : (((me >> 2) ^ 1) << 2) + (rbl - 4);
            const int lrow = (rbl < 4 ? rbl : (me & 3)) * 256;
            char* ob = rfl_ptr((char*)P.out_all[owner & 7]);
            const __amdgpu_buffer_rsrc_t ro = __builtin_amdgcn_make_buffer_rsrc(ob, 0, 1024 * NDIM * 4, 0x00020000);
            const unsigned o0 = (unsigned)((lrow + wm * 64 + fr) * NDIM + cg * 128 + wn * 64 + fq * 4) * 4u;
            const float poison = isbad ? __builtin_nanf("") : 0.0f;
#pragma unroll
            for (int m = 0; m < 4; ++m)
#pragma unroll
                for (int n = 0; n < 4; ++n)
#pragma unroll
                    for (int j = 0; j < 4; ++j) acc[m][n][j] = gelu_tanh(__builtin_amdgcn_ldexpf(acc[m][n][j], -E)) + poison;
            if (rbl < 4) {
#pragma unroll
                for (int m = 0; m < 4; ++m)
#pragma unroll
                    for (int n = 0; n < 4; ++n)
                        __builtin_amdgcn_raw_buffer_store_b128(__builtin_bit_cast(u4, acc[m][n]), ro, (int)(o0 + (unsigned)(m * 16 * NDIM * 4 + n * 64)), 0, 0);
            } else {
#pragma unroll
                for (int m = 0; m < 4; ++m)
#pragma unroll
                    for (int n = 0; n < 4; ++n)
                        __builtin_amdgcn_raw_buffer_store_b128(__builtin_bit_cast(u4, acc[m][n]), ro, (int)(o0 + (unsigned)(m * 16 * NDIM * 4 + n * 64)), 0, 17);
            }
            if (rbl >= 4) {
                WAIT_V(0);
                __syncthreads();
                if (tid == 0) {
                    char* wo = (char*)P.ws_all[owner & 7];
                    st_sys((unsigned*)(wo + OFF_DONE) + (me & 3) * 32 + g, call);
                }
            }
        }
        STAMP(ws, 3 + 2 * ntile_done);
        ++ntile_done;
    }
}

}

extern "C" void kernel_launch(int rank, int nranks, ncclComm_t comm,
                              hipStream_t stream,
                              void* const* d_in, const int* in_sizes, int n_in,
                              void* d_out, int out_size,
                              const int* in_dtypes, int out_dtype,
                              void* const* d_out_all,
                              void* const* d_ws_all, size_t ws_bytes) {
    (void)comm; (void)in_sizes; (void)n_in; (void)d_out; (void)out_size; (void)in_dtypes; (void)out_dtype;
    if (nranks != NR || (unsigned)rank >= (unsigned)NR || ws_bytes < (size_t)WS_USED) {
        fprintf(stderr, "kernel_launch: unsupported configuration (nranks %d, ws %zu)\n", nranks, ws_bytes);
        return;
    }
    HIP_CHECK(hipFuncSetAttribute((const void*)fused_gemm_gelu, hipFuncAttributeMaxDynamicSharedMemorySize, LDS_BYTES));
    Params P;
    P.x = (const float*)d_in[0]; P.w = (const float*)d_in[1];
    P.ws_all = d_ws_all; P.out_all = d_out_all; P.rank = rank;
    P.ratio8 = SCH_RATIO8; P.lead8 = SCH_LEAD8; P.xo8 = SCH_XO8; P.xor8 = SCH_XOR8;
    hipLaunchKernelGGL(fused_gemm_gelu, dim3(GRID), dim3(512), LDS_BYTES, stream, P);
    HIP_CHECK(hipGetLastError());
}
```

```cpp
#include <hip/hip_runtime.h>
#include <rccl/rccl.h>
#include <cstdio>
#include <cstdint>
#include <type_traits>

#define HIP_CHECK(expr) do { hipError_t _e = (expr); if (_e != hipSuccess) { \
    fprintf(stderr, "HIP error %s at %s:%d\n", hipGetErrorString(_e), __FILE__, __LINE__); \
    return; } } while (0)

#define SCH_RATIO8 24
#define SCH_LEAD8 72
#define SCH_XO8 420
#define SCH_OUT8 1000
#define SCH_OUTR8 8
#define SCH_XOC8 160
#define SCH_XOR8 4
#define SCH_XORC8 30

namespace {

typedef _Float16 h8 __attribute__((ext_vector_type(8)));
typedef float f4 __attribute__((ext_vector_type(4)));
typedef unsigned u4 __attribute__((ext_vector_type(4)));
#define LAS __attribute__((address_space(3)))

constexpr int NR = 8;
constexpr int NDIM = 8192;
constexpr int KSH = 1024;
#define NREL_V 2
constexpr int NREL = NREL_V;
#define NPUSH_WG 28
#define PUSH_LIM 8
constexpr int NPUSH = NPUSH_WG;
static_assert(NPUSH % 7 == 0 && NPUSH <= 32, "pusher count");
constexpr int NCH_IN = 128 + 256 + NREL * 32;
constexpr int NCH_XI = 128 + (8 - NREL) * 32 + NREL * 8;
constexpr int NCH_X = NCH_XI + 128;
constexpr int GRID = 256;
constexpr int NGW = GRID - NPUSH;
constexpr int NCONV = 768;
constexpr unsigned long long DEADLINE_TICKS = 20000000ull;

constexpr unsigned OFF_TICKET = 0, OFF_TMO = 128, OFF_TQ = 256;
constexpr unsigned OFF_LKT = 1024;
constexpr unsigned OFF_LXF = 4096;
constexpr unsigned OFF_LWF = 8192;
constexpr unsigned OFF_LOF = 12288;
constexpr unsigned OFF_DONE = 16384;
constexpr unsigned OFF_XF = 65536;
constexpr unsigned OFF_WF = 65536 + 4096;
constexpr unsigned OFF_DBG = 262144;
constexpr unsigned MIB = 1u << 20;
constexpr unsigned OFF_XL = 1 * MIB;
constexpr unsigned OFF_WL = 17 * MIB;
constexpr unsigned OFF_XR = 33 * MIB;
constexpr unsigned OFF_WR = 65 * MIB;
constexpr unsigned PAR_STRIDE = 96 * MIB;
constexpr unsigned OFF_OS = 33 * MIB + 2 * PAR_STRIDE;
constexpr unsigned WS_USED = OFF_OS + 16 * MIB;

constexpr int STAGE_B = 49152;
constexpr int L_CTL = 3 * STAGE_B;
constexpr int L_Q = L_CTL;
constexpr int L_PF = L_CTL + 1024;
constexpr int L_RED = L_CTL + 1792;
constexpr int L_BC = L_CTL + 1856;
constexpr int L_META = L_CTL + 1920;
constexpr int LDS_BYTES = L_CTL + 2048;

#define GLDS_AUX 16

struct ConvTab { unsigned short v[NCONV]; };
constexpr int conv_key(int id) {
    const int b = id / 96, r = id % 96;
    int k = r < 32 ? b * SCH_RATIO8 - 4 : ((r - 32) >> 1) * 8 + b * SCH_RATIO8 - (b >= 8 - NREL ? SCH_LEAD8 : 0);
    k += 64;
    return k < 0 ? 0 : (k > 2047 ? 2047 : k);
}
constexpr ConvTab make_conv_tab() {
    ConvTab t{}; int cnt[2049] = {};
    for (int i = 0; i < NCONV; ++i) cnt[conv_key(i) + 1]++;
    for (int k = 0; k < 2048; ++k) cnt[k + 1] += cnt[k];
    for (int i = 0; i < NCONV; ++i) t.v[cnt[conv_key(i)]++] = (unsigned short)i;
    return t;
}
__constant__ const ConvTab conv_tab = make_conv_tab();

constexpr int chunk_key(int id, bool ing) {
    int k = 0;
    const int nW = ing ? 256 : (8 - NREL) * 32, r = id - 128, q = r - nW;
    if (!ing && id >= NCH_XI) k = SCH_OUT8 + (id - NCH_XI);
    else if (id < 128) { const int b = id >> 4, rbl = (id >> 1) & 7; k = rbl >= 4 ? b * SCH_RATIO8 - 4 : (ing ? SCH_XO8 + (b * 8 + (id & 7)) * SCH_XOR8 / 8 : SCH_XOC8 + (b * 8 + (id & 7)) * SCH_XORC8 / 8); }
    else if (r < nW) k = (r & 31) * 8 + (r >> 5) * SCH_RATIO8;
    else if (ing) k = (q & 7) * 32 + 12 + (8 - NREL + (q >> 5)) * SCH_RATIO8;
    else k = (q & 7) * 32 + 12 + (8 - NREL + (q >> 3)) * SCH_RATIO8 - SCH_LEAD8;
    k += 64;
    return k < 0 ? 0 : (k > 2047 ? 2047 : k);
}
struct PushTab { unsigned v[2][512]; };
constexpr PushTab make_push_tab() {
    PushTab t{};
    for (int c = 0; c < 2; ++c) {
        const int n = c == 0 ? NCH_IN : NCH_X; int cnt[2049] = {};
        for (int i = 0; i < n; ++i) cnt[chunk_key(i, c == 0) + 1]++;
        for (int k = 0; k < 2048; ++k) cnt[k + 1] += cnt[k];
        for (int i = 0; i < n; ++i) t.v[c][cnt[chunk_key(i, c == 0)]++] = (unsigned)i;
    }
    return t;
}
__constant__ const PushTab push_tab = make_push_tab();

struct Params {
    const float* x; const float* w;
    void* const* ws_all; void* const* out_all;
    int rank;
    int ratio8, lead8, xo8, xor8;
};

#define GAS __attribute__((address_space(1)))
__device__ __forceinline__ unsigned ld_sys(const unsigned* p) { return __hip_atomic_load((const GAS unsigned*)p, __ATOMIC_RELAXED, __HIP_MEMORY_SCOPE_SYSTEM); }
__device__ __forceinline__ unsigned ld_agent(const unsigned* p) { return __hip_atomic_load((const GAS unsigned*)p, __ATOMIC_RELAXED, __HIP_MEMORY_SCOPE_AGENT); }
__device__ __forceinline__ void st_sys(unsigned* p, unsigned v) { __hip_atomic_store((GAS unsigned*)p, v, __ATOMIC_RELAXED, __HIP_MEMORY_SCOPE_SYSTEM); }
__device__ __forceinline__ void st_agent(unsigned* p, unsigned v) { __hip_atomic_store((GAS unsigned*)p, v, __ATOMIC_RELAXED, __HIP_MEMORY_SCOPE_AGENT); }
__device__ __forceinline__ unsigned add_agent(unsigned* p) { return __hip_atomic_fetch_add((GAS unsigned*)p, 1u, __ATOMIC_RELAXED, __HIP_MEMORY_SCOPE_AGENT); }
__device__ __forceinline__ int rfl(int v) { return __builtin_amdgcn_readfirstlane(v); }
__device__ __forceinline__ unsigned rflu(unsigned v) { return (unsigned)__builtin_amdgcn_readfirstlane((int)v); }
__device__ __forceinline__ char* rfl_ptr(char* p) {
    const unsigned long long v = (unsigned long long)p;
    return (char*)(((unsigned long long)rflu((unsigned)(v >> 32)) << 32) | rflu((unsigned)v));
}
#define WAIT_V(n) asm volatile("s_waitcnt vmcnt(" #n ")" ::: "memory")
#define WAIT_L0() asm volatile("s_waitcnt lgkmcnt(0)" ::: "memory")
#define BAR() do { asm volatile("" ::: "memory"); __builtin_amdgcn_s_barrier(); asm volatile("" ::: "memory"); } while (0)

#define STAMP(ws, slot) do { } while (0)

__device__ __forceinline__ int rb32_of(int t, int rbl) {
    return rbl < 4 ? 4 * t + rbl : 4 * ((((t >> 2) ^ 1) << 2) + (rbl - 4)) + (t & 3);
}

__device__ __forceinline__ int block_scale(float am, LAS unsigned char* lds, int par, int& e) {
#pragma unroll
    for (int o = 32; o >= 1; o >>= 1) am = fmaxf(am, __shfl_xor(am, o));
    LAS float* red = (LAS float*)(lds + L_RED) + par * 8;
    if ((threadIdx.x & 63) == 0) red[threadIdx.x >> 6] = am;
    __syncthreads();
    float m = red[0];
#pragma unroll
    for (int i = 1; i < 8; ++i) m = fmaxf(m, red[i]);
    const int ex = __builtin_amdgcn_frexp_expf(m);
    e = rfl((15 - ex) & ~3);
    return rfl(m == 0.0f ? 0 : e + 257);
}
__device__ __forceinline__ float amax4(f4 v, float a) {
    return fmaxf(fmaxf(fmaxf(fabsf(v[0]), fabsf(v[1])), fmaxf(fabsf(v[2]), fabsf(v[3]))), a);
}
__device__ __forceinline__ _Float16 cvt1(float v, int e) { return (_Float16)__builtin_amdgcn_ldexpf(v, e); }

__device__ __forceinline__ float gelu_tanh(float y) {
    const float u = 0.7978845608028654f * (y + 0.044715f * y * y * y);
    const float ex = __expf(-2.0f * u);
    return y * __builtin_amdgcn_rcpf(1.0f + ex);
}

__global__ void __launch_bounds__(512, 2) fused_gemm_gelu(Params P) {
    extern __shared__ __attribute__((aligned(16))) unsigned char smem[];
    LAS unsigned char* lds = (LAS unsigned char*)smem;
    const int tid = threadIdx.x, lane = tid & 63, wid = rfl(tid >> 6);
    const int me = P.rank;
    char* ws = rfl_ptr((char*)P.ws_all[me]);
    unsigned* tmo = (unsigned*)(ws + OFF_TMO);

    if (tid == 0) {
        const unsigned t = add_agent((unsigned*)(ws + OFF_TICKET));
        *(LAS unsigned*)(lds + L_BC) = t / GRID + 1u;
    }
    __syncthreads();
    const unsigned call = rflu(*(LAS unsigned*)(lds + L_BC));
    const unsigned seq = call & 0x3FFFFFu;
    const unsigned par = call & 1u;
    const unsigned xf_off = OFF_XF + par * 65536u, wf_off = OFF_WF + par * 65536u;
    const unsigned xr_off = OFF_XR + par * PAR_STRIDE, wr_off = OFF_WR + par * PAR_STRIDE;
    const int bi = blockIdx.x;
    STAMP(ws, 0);

    if (bi < NPUSH) {
        bool dead = false;
        const unsigned loff = (unsigned)lane * 16u;
        u4 buf[4][4];
#pragma unroll
        for (int j = 0; j < 4; ++j)
#pragma unroll
            for (int i = 0; i < 4; ++i) buf[j][i] = (u4){0u, 0u, 0u, 0u};
        const int b7 = bi % 7, ci = b7 < 3 ? (bi / 7) * 3 + b7 : (bi / 7) * 4 + b7 - 3;
        if (wid < 4) {
        const int lx = b7 < 3 ? (ci * 4 + wid) % 3 : 3 + ((wid + ci) & 3);
        const int t = lx < 3 ? (me & 4) + ((me + 1 + lx) & 3) : ((me & 4) ^ 4) + ((me + lx) & 3);
        const bool ingrp = lx < 3;
        const int nch = ingrp ? NCH_IN : NCH_X;
        unsigned* tkp = (unsigned*)(ws + OFF_LKT + lx * 128);
        const unsigned tkbase = (call - 1u) * (unsigned)(nch + 4 * NPUSH / 7);
        char* wsd = rfl_ptr((char*)P.ws_all[t & 7]);
        const unsigned* ord = push_tab.v[ingrp ? 0 : 1];
        char* outd = rfl_ptr((char*)P.out_all[t & 7]);
        struct Item { unsigned src, dst, wfl, dfl, out; };
        auto decode = [&](int tk) {
            const int id = (int)ord[tk];
            Item it; it.out = 0u;
            if (!ingrp && id >= NCH_XI) {
                const int q = id - NCH_XI, sg = q >> 4, c = q & 15;
                it.src = OFF_OS + (unsigned)((t & 3) * 128 + q) * 32768u; it.wfl = OFF_LOF + (unsigned)((t & 3) * 32 + sg * 4) * 4u;
                it.dst = (unsigned)(((me & 3) * 256 + c * 16) * NDIM + (me >> 2) * 4096 + sg * 512) * 4u;
                it.dfl = OFF_DONE + (unsigned)((me & 3) * 128 + q) * 4u; it.out = 1u;
            } else if (id < 128) {
                const int b = id >> 4, rbl = (id >> 1) & 7, ks = id & 1, rb = rb32_of(t, rbl);
                it.src = OFF_XL + (unsigned)(rb * 8 + b) * 65536u + ks * 32768u; it.wfl = OFF_LXF + (unsigned)(rb * 8 + b) * 4u;
                it.dst = xr_off + (unsigned)((rbl * 8 + me) * 8 + b) * 65536u + ks * 32768u;
                it.dfl = xf_off + (unsigned)((rbl * 2 + ks) * 64 + me * 8 + b) * 4u;
            } else {
                const int nW = ingrp ? 256 : (8 - NREL) * 32, r = id - 128, q = r - nW;
                if (r < nW || !ingrp) {
                    int b = r >> 5, g = r & 31;
                    if (r >= nW) { b = 8 - NREL + (q >> 3); g = 4 * (q & 7) + (((t & 3) - b) & 3); }
                    const int cg = (t >> 2) * 32 + g;
                    it.src = OFF_WL + (unsigned)(cg * 8 + b) * 32768u; it.wfl = OFF_LWF + (unsigned)(cg * 8 + b) * 4u;
                    it.dst = wr_off + (unsigned)((g * 8 + me) * 8 + b) * 32768u; it.dfl = wf_off + (unsigned)(g * 64 + me * 8 + b) * 4u;
                } else {
                    const int b = 8 - NREL + (q >> 5), ur = (((me >> 2) ^ 1) << 2) + ((q >> 3) & 3), g = 4 * (q & 7) + (((me & 3) - b) & 3);
                    it.src = it.dst = wr_off + (unsigned)((g * 8 + ur) * 8 + b) * 32768u;
                    it.wfl = it.dfl = wf_off + (unsigned)(g * 64 + ur * 8 + b) * 4u;
                }
            }
            return it;
        };
        auto draw = [&]() {
            unsigned v = 0;
            if (lane == 0) v = add_agent(tkp);
            return v;
        };
        int tk = (int)(rflu(draw()) - tkbase);
        Item cur{0u, 0u, 0u, 0u, 0u}, prv{0u, 0u, 0u, 0u, 0u}; unsigned cur_val = 0u, prv_val = 0u; bool cur_ok = false, prv_ok = false;
        unsigned fv = 0u;
        bool have = tk < nch;
        Item nxt{0u, 0u, 0u, 0u, 0u};
        if (have) nxt = decode(tk);
        for (;;) {
            cur_ok = false;
            if (have) {
                if (!prv_ok) {
                    const unsigned long long t0 = __builtin_amdgcn_s_memrealtime();
                    for (;;) {
                        const unsigned pv = ld_sys((const unsigned*)(ws + nxt.wfl + nxt.out * (unsigned)(lane & 3) * 4u));
                        fv = __all((pv >> 10) == seq) ? rflu(pv) : 0u;
                        if ((fv >> 10) == seq) break;
                        if (rflu(ld_agent(tmo)) == call || __builtin_amdgcn_s_memrealtime() - t0 > DEADLINE_TICKS) {
                            if (lane == 0) st_agent(tmo, call);
                            dead = true; break;
                        }
                        __builtin_amdgcn_s_sleep(2);
                    }
                    if (dead) break;
                }
                if ((fv >> 10) == seq) { cur = nxt; cur_val = fv; cur_ok = true; }
            } else if (!prv_ok) break;
            const __amdgpu_buffer_rsrc_t rl = __builtin_amdgcn_make_buffer_rsrc(ws, 0, cur_ok ? (int)WS_USED : 0, 0x00020000);
            const __amdgpu_buffer_rsrc_t rsp = __builtin_amdgcn_make_buffer_rsrc(prv.out ? outd : wsd, 0, prv_ok ? (prv.out ? 1024 * NDIM * 4 : (int)WS_USED) : 0, 0x00020000);
            const __amdgpu_buffer_rsrc_t rsc = __builtin_amdgcn_make_buffer_rsrc(cur.out ? outd : wsd, 0, cur_ok ? (cur.out ? 1024 * NDIM * 4 : (int)WS_USED) : 0, 0x00020000);
            const unsigned pks = prv.out ? NDIM * 4u : 2048u, cks = cur.out ? NDIM * 4u : 2048u;
            const unsigned pdo = prv.dst + loff + 8u * pks, cdo = cur.dst + loff;
            unsigned tkv = 0u, fvv = 0u;
            if (cur_ok && lane == 0) {
                const unsigned one = 1u;
                asm volatile("global_atomic_add %0, %1, %2, off sc0" : "=v"(tkv) : "v"(tkp), "v"(one) : "memory");
            }
#pragma unroll
            for (int st = 0; st < 8; ++st) {
                const int j = st & 3;
                if (st == 4) {
                    asm volatile("s_waitcnt vmcnt(32)" : "+v"(tkv) :: "memory");
                    if (cur_ok) { tk = (int)(rflu(tkv) - tkbase); have = tk < nch; if (have) nxt = decode(tk); }
                    fvv = ld_sys((const unsigned*)(ws + nxt.wfl + nxt.out * (unsigned)(lane & 3) * 4u));
                }
                if (st == 5) {
                    WAIT_V(10);
                    if (prv_ok && lane == 0) st_sys((unsigned*)(wsd + prv.dfl), prv_val);
                }
#pragma unroll
                for (int i = 0; i < 4; ++i) {
                    asm volatile("s_waitcnt vmcnt(%0)" :: "n"(PUSH_LIM) : "memory");
                    if (st < 4) __builtin_amdgcn_raw_buffer_store_b128(buf[j][i], rsp, (int)(pdo + (unsigned)(st * 2 + (i >> 1)) * pks + (unsigned)(i & 1) * 1024u), 0, 17);
                    else __builtin_amdgcn_raw_buffer_store_b128(buf[j][i], rsc, (int)(cdo + (unsigned)((st - 4) * 2 + (i >> 1)) * cks + (unsigned)(i & 1) * 1024u), 0, 17);
                    buf[j][i] = __builtin_amdgcn_raw_buffer_load_b128(rl, (int)(cur.src + loff + (unsigned)(st * 4096 + i * 1024)), 0, 16);
                }
            }
            fv = __all((fvv >> 10) == seq) ? rflu(fvv) : 0u;
            prv = cur; prv_val = cur_val; prv_ok = cur_ok;
        }
        }
        if (bi == 0 && wid == 4) {
            const unsigned* dp = (const unsigned*)(ws + OFF_DONE);
            const unsigned long long t0 = __builtin_amdgcn_s_memrealtime();
            bool bad = dead;
            while (!bad) {
                bool ok = true;
#pragma unroll
                for (int i = 0; i < 8; ++i) ok = ok && (ld_sys(dp + i * 64 + lane) >> 10) == seq;
                if (__all(ok)) break;
                if (rflu(ld_agent(tmo)) == call || __builtin_amdgcn_s_memrealtime() - t0 > DEADLINE_TICKS) {
                    if (lane == 0) st_agent(tmo, call);
                    bad = true; break;
                }
                __builtin_amdgcn_s_sleep(2);
            }
            __builtin_amdgcn_fence(__ATOMIC_ACQUIRE, "");
            if (bad && lane == 0) {
                float* o = (float*)P.out_all[me];
                o[1023 * (size_t)NDIM + (((me >> 2) ^ 1) * 4096 + 4095)] = __builtin_nanf("");
            }
        }
        return;
    }

    {
        const __amdgpu_buffer_rsrc_t rsl = __builtin_amdgcn_make_buffer_rsrc(ws, 0, (int)WS_USED, 0x00020000);
        int rp = 0;
        for (int it = bi - NPUSH; it < NCONV; it += NGW, rp ^= 1) {
            const int id = conv_tab.v[it], b = id / 96, r = id % 96;
            int e, code; unsigned fo;
            if (r < 32) {
                const int rb = r;
                f4 v[16]; float am = 0.f;
#pragma unroll
                for (int i = 0; i < 8; ++i) {
                    const int q = i * 512 + tid, ks = q >> 11, row = (q >> 3) & 255, c = (q & 7) ^ (row & 7);
                    const float* s = P.x + (size_t)(rb * 256 + row) * KSH + b * 128 + ks * 64 + c * 8;
                    v[2 * i] = *(const f4*)s; v[2 * i + 1] = *(const f4*)(s + 4);
                }
#pragma unroll
                for (int i = 0; i < 16; ++i) am = amax4(v[i], am);
                code = block_scale(am, lds, rp, e);
                const unsigned base = OFF_XL + (unsigned)(rb * 8 + b) * 65536u + (unsigned)tid * 16u;
#pragma unroll
                for (int i = 0; i < 8; ++i) {
                    h8 o;
#pragma unroll
                    for (int j = 0; j < 4; ++j) { o[j] = cvt1(v[2 * i][j], e); o[4 + j] = cvt1(v[2 * i + 1][j], e); }
                    __builtin_amdgcn_raw_buffer_store_b128(__builtin_bit_cast(u4, o), rsl, (int)(base + i * 8192u), 0, 17);
                }
                fo = OFF_LXF + (unsigned)(rb * 8 + b) * 4u;
            } else {
                const int g2 = r - 32, cg = (g2 >> 1) + (g2 & 1) * 32;
                const int n4 = tid & 31, kc = tid >> 5;
                f4 v[8]; float am = 0.f;
#pragma unroll
                for (int j = 0; j < 8; ++j) v[j] = *(const f4*)(P.w + (size_t)(b * 128 + kc * 8 + j) * NDIM + cg * 128 + n4 * 4);
#pragma unroll
                for (int j = 0; j < 8; ++j) am = amax4(v[j], am);
                code = block_scale(am, lds, rp, e);
#pragma unroll
                for (int e4 = 0; e4 < 4; ++e4) {
                    const int n = n4 * 4 + e4; h8 o;
#pragma unroll
                    for (int j = 0; j < 8; ++j) o[j] = cvt1(v[j][e4], e);
                    *(LAS h8*)(lds + (kc >> 3) * 16384 + n * 128 + (((kc & 7) ^ (n & 7)) << 4)) = o;
                }
                __syncthreads();
                const unsigned base = OFF_WL + (unsigned)(cg * 8 + b) * 32768u + (unsigned)tid * 16u;
#pragma unroll
                for (int i = 0; i < 4; ++i) {
                    const u4 d = *(LAS u4*)(lds + i * 8192 + tid * 16);
                    __builtin_amdgcn_raw_buffer_store_b128(d, rsl, (int)(base + i * 8192u), 0, 17);
                }
                fo = OFF_LWF + (unsigned)(cg * 8 + b) * 4u;
            }
            WAIT_V(0);
            __syncthreads();
            if (tid == 0) st_sys((unsigned*)(ws + fo), (seq << 10) | (unsigned)code);
        }
    }
    STAMP(ws, 1);

    const int wm = wid >> 1, wn = wid & 1, fr = lane & 15, fq = lane >> 4;
    const int fo0 = fr * 128 + ((fq ^ (fr & 7)) << 4), fo1 = fo0 ^ 64;
    const int abase = wm * 8192, bbase = 32768 + wn * 8192;
    constexpr unsigned TQ_PER_CALL = 256u + (32u - NPUSH);
    int xcd = bi & 7;
    int u = (bi >> 3) - ((NPUSH - xcd + 7) >> 3);
    int ntile_done = 0;
    bool pool = u >= 28;
    for (;;) {
        if (pool) {
            if (tid == 0) {
                const unsigned o = add_agent((unsigned*)(ws + OFF_TQ));
                *(LAS unsigned*)(lds + L_BC + 4) = o - (call - 1u) * TQ_PER_CALL;
            }
            __syncthreads();
            const unsigned nx = rflu(*(LAS unsigned*)(lds + L_BC + 4));
            if (nx >= 32u) break;
            xcd = (int)(nx >> 2); u = 28 + (int)(nx & 3u);
        }
        pool = true;
        const int g = 4 * xcd + ((u & 15) >> 2);
        const int rbl = (u < 16 ? 4 : 0) + (u & 3);
        const int rb = rb32_of(me, rbl), cg = (me >> 2) * 32 + g;
        f4 acc[4][4];
#pragma unroll
        for (int m = 0; m < 4; ++m)
#pragma unroll
            for (int n = 0; n < 4; ++n) acc[m][n] = (f4){0.f, 0.f, 0.f, 0.f};
        const int ls = lane >> 3, lb = lane & 7;
        const unsigned* xfp = (const unsigned*)(ws + (ls == me ? OFF_LXF + (unsigned)(rb * 8 + lb) * 4u : xf_off + (unsigned)(rbl * 128 + lane) * 4u));
        const unsigned* xgp = (const unsigned*)(ws + (ls == me ? OFF_LXF + (unsigned)(rb * 8 + lb) * 4u : xf_off + (unsigned)(rbl * 128 + 64 + lane) * 4u));
        const unsigned* wfp = (const unsigned*)(ws + (ls == me ? OFF_LWF + (unsigned)(cg * 8 + lb) * 4u : wf_off + (unsigned)(g * 64 + lane) * 4u));
        const unsigned blk_a = ls == me ? OFF_XL + (unsigned)(rb * 8 + lb) * 65536u : xr_off + (unsigned)(rbl * 64 + lane) * 65536u;
        const unsigned blk_b = ls == me ? OFF_WL + (unsigned)(cg * 8 + lb) * 32768u : wr_off + (unsigned)(g * 64 + lane) * 32768u;
        unsigned long long seen = 0ull; int nq = 0, Ecur = 0, Emin = 0, haveE = 0, bad = 0;
        int polled = 0;
        int cs = 0, is = 0, avail = 0, ep = 0, allk = 0; unsigned qa = 0u, qb = 0u, meta = 0u, metav = 0u;
        int rd = 0, gst = 0;

        auto absorb = [&](unsigned xv, unsigned xw, unsigned wv) {
            unsigned long long rdy = __ballot((xv >> 10) == seq && (xw >> 10) == seq && (wv >> 10) == seq) & ~seen;
            seen |= rdy;
            if (rdy == 0ull) return;
            LAS unsigned* q = (LAS unsigned*)(lds + L_Q);
            const int cx = (int)(xv & 1023u), cw = (int)(wv & 1023u), el = cx + cw - 514;
            if (!haveE) {
                const unsigned long long nz = __ballot(cx != 0 && cw != 0) & rdy;
                if (nz) { haveE = 1; Ecur = __builtin_amdgcn_readlane(el, __builtin_ctzll(nz)); Emin = Ecur; }
            }
            const unsigned long long same = __ballot(cx != 0 && cw != 0 && el == Ecur) & rdy;
            if ((same >> lane) & 1ull) {
                const int pos = nq + __builtin_popcountll(same & ((1ull << lane) - 1ull));
                q[pos] = blk_a; q[64 + pos] = blk_b;
            }
            nq += __builtin_popcountll(same);
            rdy &= ~same;
            while (rdy) {
                const int l = __builtin_ctzll(rdy); rdy &= rdy - 1;
                if (__builtin_amdgcn_readlane(cx, l) == 0 || __builtin_amdgcn_readlane(cw, l) == 0) continue;
                const int e = __builtin_amdgcn_readlane(el, l);
                if (e > Emin + 60) continue;
                const int delta = e - Ecur; Ecur = e; Emin = min(Emin, e);
                if (lane == l) { q[nq] = blk_a | ((unsigned)delta & 0xFFFFu); q[64 + nq] = blk_b; }
                ++nq;
            }
        };
        auto publish = [&]() {
            const bool all = (seen == ~0ull) || bad;
            ((LAS unsigned*)(lds + L_META))[ep & 1] = (unsigned)nq | (all ? 128u : 0u) | (bad ? 256u : 0u) | ((unsigned)(Ecur + 1024) << 16);
        };
        auto fetch_req = [&]() {
            LAS unsigned* q = (LAS unsigned*)(lds + L_Q);
            qa = q[lane]; qb = q[64 + lane];
            metav = ((LAS unsigned*)(lds + L_META))[ep & 1];
            ++ep;
        };
        auto fetch_use = [&]() { meta = rflu(metav); avail = 2 * (int)(meta & 127u); allk = (int)((meta >> 7) & 1u); };
        const char* ga = nullptr; const char* gb = nullptr; int gdst = 0;
        const char* wsa = ws + wid * 4096 + lane * 16; const char* wsb = ws + wid * 2048 + lane * 16;
        auto issue_addr = [&]() {
            const unsigned ks = (unsigned)is & 1u;
            ga = wsa + (((unsigned)__builtin_amdgcn_readlane((int)qa, is >> 1) & 0xFFFF0000u) + ks * 32768u);
            gb = wsb + ((unsigned)__builtin_amdgcn_readlane((int)qb, is >> 1) + ks * 16384u);
            gdst = gst; gst = gst == 2 * STAGE_B ? 0 : gst + STAGE_B;
            ++is;
        };
#define GLDS(gp, lp, off) __builtin_amdgcn_global_load_lds((const unsigned*)(gp), (LAS unsigned*)(lp), 16, off, GLDS_AUX)
        auto issue_a = [&](int h) {
            if (h == 0) { GLDS(ga, lds + gdst + wid * 4096, 0); GLDS(ga, lds + gdst + wid * 4096, 1024); }
            else { GLDS(ga, lds + gdst + wid * 4096, 2048); GLDS(ga, lds + gdst + wid * 4096, 3072); }
        };
        auto issue_b = [&](int) { GLDS(gb, lds + gdst + 32768 + wid * 2048, 0); GLDS(gb, lds + gdst + 32768 + wid * 2048, 1024); };
        auto issue = [&]() { issue_addr(); issue_a(0); issue_a(1); issue_b(0); };
        h8 a0[4], b0[4], a1[4], b1[4];
        auto read_frags = [&](h8 (&af)[4], h8 (&bf)[4], int st, int fo) {
            LAS unsigned char* sp = lds + st + fo;
#pragma unroll
            for (int n = 0; n < 4; ++n) bf[n] = *(LAS h8*)(sp + bbase + n * 2048);
#pragma unroll
            for (int m = 0; m < 4; ++m) af[m] = *(LAS h8*)(sp + abase + m * 2048);
        };
#define MMA_ROW(m, af, bf) do { _Pragma("unroll") for (int n = 0; n < 4; ++n) \
            asm volatile("v_mfma_f32_16x16x32_f16 %0, %1, %2, %0" : "+v"(acc[m][n]) : "v"(bf[n]), "v"(af[m])); } while (0)
#define SCHED() __builtin_amdgcn_sched_barrier(0)
#define ACC_FENCE() do { SCHED(); asm volatile("s_nop 15\n\ts_nop 15" : "+v"(acc[0][0]), "+v"(acc[0][1]), "+v"(acc[0][2]), "+v"(acc[0][3]), \
            "+v"(acc[1][0]), "+v"(acc[1][1]), "+v"(acc[1][2]), "+v"(acc[1][3]), "+v"(acc[2][0]), "+v"(acc[2][1]), "+v"(acc[2][2]), "+v"(acc[2][3]), \
            "+v"(acc[3][0]), "+v"(acc[3][1]), "+v"(acc[3][2]), "+v"(acc[3][3])); SCHED(); } while (0)
        int pre = 0;
        auto step = [&]() {
            __builtin_amdgcn_s_waitcnt(0xC07F);
            MMA_ROW(0, a0, b0);
            SCHED();
            read_frags(a1, b1, rd, fo1);
            SCHED();
            MMA_ROW(1, a0, b0);
            SCHED();
            const int upd = allk ^ 1;
            if (upd && wid == 0) {
                if (polled == 2) { absorb(((LAS unsigned*)(lds + L_PF))[lane], ((LAS unsigned*)(lds + L_PF))[128 + lane], ((LAS unsigned*)(lds + L_PF))[64 + lane]); polled = 0; }
                publish();
            }
            SCHED();
            MMA_ROW(2, a0, b0);
            SCHED();
            pre = is > cs + 1 ? 1 : 0;
            const bool deep = is > cs + 2;
            const bool go = is < avail;
            if (go) issue_addr();
            rd = rd == 2 * STAGE_B ? 0 : rd + STAGE_B;
            SCHED();
            MMA_ROW(3, a0, b0);
            SCHED();
            if (deep) WAIT_V(6); else WAIT_V(0);
            __builtin_amdgcn_s_waitcnt(0xC07F);
            BAR();
            MMA_ROW(0, a1, b1);
            SCHED();
            if (upd) fetch_req();
            if (pre) read_frags(a0, b0, rd, fo0);
            SCHED();
            MMA_ROW(1, a1, b1);
            SCHED();
            if (upd && wid == 0) {
                if (polled == 1) polled = 2;
                else if (seen != ~0ull) {
                    __builtin_amdgcn_global_load_lds(xfp, (LAS unsigned*)(lds + L_PF), 4, 0, 17);
                    __builtin_amdgcn_global_load_lds(wfp, (LAS unsigned*)(lds + L_PF + 256), 4, 0, 17);
                    __builtin_amdgcn_global_load_lds(xgp, (LAS unsigned*)(lds + L_PF + 512), 4, 0, 17);
                    polled = 1;
                }
            }
            if (go) issue_a(0);
            SCHED();
            MMA_ROW(2, a1, b1);
            SCHED();
            if (go) issue_a(1);
            SCHED();
            MMA_ROW(3, a1, b1);
            SCHED();
            if (go) issue_b(0);
            if (upd) {
                fetch_use();
                while (is < avail && is <= cs + 3) issue();
            }
            ++cs;
        };
        auto fast_step = [&](auto first) {
            constexpr bool FIRST = decltype(first)::value;
            __builtin_amdgcn_s_waitcnt(0xC07F);
            MMA_ROW(0, a0, b0);
            SCHED();
            read_frags(a1, b1, rd, fo1);
            SCHED();
            MMA_ROW(1, a0, b0);
            SCHED();
            const int upd = allk ^ 1;
            if (FIRST && upd && wid == 0) {
                if (polled == 2) { absorb(((LAS unsigned*)(lds + L_PF))[lane], ((LAS unsigned*)(lds + L_PF))[128 + lane], ((LAS unsigned*)(lds + L_PF))[64 + lane]); polled = 0; }
                publish();
            }
            issue_addr();
            SCHED();
            MMA_ROW(2, a0, b0);
            rd = rd == 2 * STAGE_B ? 0 : rd + STAGE_B;
            MMA_ROW(3, a0, b0);
            SCHED();
            WAIT_V(6);
            __builtin_amdgcn_s_waitcnt(0xC07F);
            BAR();
            MMA_ROW(0, a1, b1);
            SCHED();
            read_frags(a0, b0, rd, fo0);
            if (FIRST && upd) {
                fetch_req();
                if (wid == 0) {
                    if (polled == 1) polled = 2;
                    else if (seen != ~0ull) {
                        __builtin_amdgcn_global_load_lds(xfp, (LAS unsigned*)(lds + L_PF), 4, 0, 17);
                        __builtin_amdgcn_global_load_lds(wfp, (LAS unsigned*)(lds + L_PF + 256), 4, 0, 17);
                        __builtin_amdgcn_global_load_lds(xgp, (LAS unsigned*)(lds + L_PF + 512), 4, 0, 17);
                        polled = 1;
                    }
                }
            }
            if (!FIRST) polled = polled == 1 ? 2 : polled;
            SCHED();
            MMA_ROW(1, a1, b1);
            SCHED();
            issue_a(0);
            SCHED();
            MMA_ROW(2, a1, b1);
            SCHED();
            issue_a(1);
            SCHED();
            MMA_ROW(3, a1, b1);
            SCHED();
            issue_b(0);
            if (FIRST && upd) fetch_use();
            ++cs;
        };
        int delta = 0;
        auto restart = [&]() {
            while (cs == is && !(allk && cs == avail)) {
                if (wid == 0) {
                    const int nq0 = nq;
                    WAIT_V(0); polled = 0;
                    const unsigned long long t0 = __builtin_amdgcn_s_memrealtime();
                    for (;;) {
                        const unsigned xv = ld_sys(xfp), xw = ld_sys(xgp), wv = ld_sys(wfp);
                        absorb(xv, xw, wv);
                        if (nq > nq0 || seen == ~0ull) break;
                        if (rflu(ld_agent(tmo)) == call || __builtin_amdgcn_s_memrealtime() - t0 > DEADLINE_TICKS) {
                            if (lane == 0) st_agent(tmo, call);
                            bad = 1; break;
                        }
                        __builtin_amdgcn_s_sleep(2);
                    }
                    publish();
                    WAIT_L0();
                }
                BAR();
                fetch_req(); fetch_use();
                while (is < avail && is <= cs + 2) issue();
            }
            pre = cs == is ? 2 : 1;
            if (pre == 1) {
                if (is > cs + 2) WAIT_V(12); else if (is > cs + 1) WAIT_V(6); else WAIT_V(0);
                BAR();
                read_frags(a0, b0, rd, fo0);
            }
        };
        auto prepare = [&]() {
            if (!pre) restart();
            delta = pre == 2 ? 0 : (int)(short)__builtin_amdgcn_readlane((int)qa, cs >> 1);
        };
        prepare();
        while (pre != 2) {
            const int dc = max(delta, -250), d1 = dc >> 1, d2 = dc - d1;
            const float s1 = __builtin_bit_cast(float, (unsigned)(127 + d1) << 23), s2 = __builtin_bit_cast(float, (unsigned)(127 + d2) << 23);
            ACC_FENCE();
#pragma unroll
            for (int m = 0; m < 4; ++m)
#pragma unroll
                for (int n = 0; n < 4; ++n) acc[m][n] = (acc[m][n] * s1) * s2;
            ACC_FENCE();
            do {
                delta = 0;
                while (is == cs + 3 && avail >= cs + 5 && delta == 0) {
                    fast_step(std::true_type{});
                    fast_step(std::false_type{});
                    delta = (int)(short)__builtin_amdgcn_readlane((int)qa, cs >> 1);
                }
                if (delta == 0) {
                    step();
                    if (!pre) restart();
                    step();
                    prepare();
                }
            } while (pre != 2 && delta == 0);
        }
        ACC_FENCE();
        STAMP(ws, 2 + 2 * ntile_done);
        {
            const int E = (int)(meta >> 16) - 1024;
            const bool isbad = (meta & 256u) != 0u || rflu(ld_agent(tmo)) == call;
            char* ob = rfl_ptr((char*)P.out_all[me]);
            const __amdgpu_buffer_rsrc_t ro = __builtin_amdgcn_make_buffer_rsrc(ob, 0, 1024 * NDIM * 4, 0x00020000);
            const __amdgpu_buffer_rsrc_t rw = __builtin_amdgcn_make_buffer_rsrc(ws, 0, (int)WS_USED, 0x00020000);
            const unsigned o0 = (unsigned)(((rbl & 3) * 256 + wm * 64 + fr) * NDIM + cg * 128 + wn * 64 + fq * 4) * 4u;
            const unsigned o1 = OFF_OS + (unsigned)((rbl & 3) * 8 + (g >> 2)) * 524288u + (unsigned)((wm * 64 + fr) * 512 + (g & 3) * 128 + wn * 64 + fq * 4) * 4u;
            const float poison = isbad ? __builtin_nanf("") : 0.0f;
#pragma unroll
            for (int m = 0; m < 4; ++m)
#pragma unroll
                for (int n = 0; n < 4; ++n)
#pragma unroll
                    for (int j = 0; j < 4; ++j) acc[m][n][j] = gelu_tanh(__builtin_amdgcn_ldexpf(acc[m][n][j], -E)) + poison;
            if (rbl < 4) {
#pragma unroll
                for (int m = 0; m < 4; ++m)
#pragma unroll
                    for (int n = 0; n < 4; ++n)
                        __builtin_amdgcn_raw_buffer_store_b128(__builtin_bit_cast(u4, acc[m][n]), ro, (int)(o0 + (unsigned)(m * 16 * NDIM * 4 + n * 64)), 0, 0);
            } else {
#pragma unroll
                for (int m = 0; m < 4; ++m)
#pragma unroll
                    for (int n = 0; n < 4; ++n)
                        __builtin_amdgcn_raw_buffer_store_b128(__builtin_bit_cast(u4, acc[m][n]), rw, (int)(o1 + (unsigned)(m * 16 * 2048 + n * 64)), 0, 17);
            }
            if (rbl >= 4) {
                WAIT_V(0);
                __syncthreads();
                if (tid == 0) st_sys((unsigned*)(ws + OFF_LOF) + (rbl & 3) * 32 + g, seq << 10);
            }
        }
        STAMP(ws, 3 + 2 * ntile_done);
        ++ntile_done;
    }
}

}

extern "C" void kernel_launch(int rank, int nranks, ncclComm_t comm,
                              hipStream_t stream,
                              void* const* d_in, const int* in_sizes, int n_in,
                              void* d_out, int out_size,
                              const int* in_dtypes, int out_dtype,
                              void* const* d_out_all,
                              void* const* d_ws_all, size_t ws_bytes) {
    (void)comm; (void)in_sizes; (void)n_in; (void)d_out; (void)out_size; (void)in_dtypes; (void)out_dtype;
    if (nranks != NR || (unsigned)rank >= (unsigned)NR || ws_bytes < (size_t)WS_USED) {
        fprintf(stderr, "kernel_launch: unsupported configuration (nranks %d, ws %zu)\n", nranks, ws_bytes);
        return;
    }
    HIP_CHECK(hipFuncSetAttribute((const void*)fused_gemm_gelu, hipFuncAttributeMaxDynamicSharedMemorySize, LDS_BYTES));
    Params P;
    P.x = (const float*)d_in[0]; P.w = (const float*)d_in[1];
    P.ws_all = d_ws_all; P.out_all = d_out_all; P.rank = rank;
    P.ratio8 = SCH_RATIO8; P.lead8 = SCH_LEAD8; P.xo8 = SCH_XO8; P.xor8 = SCH_XOR8;
    hipLaunchKernelGGL(fused_gemm_gelu, dim3(GRID), dim3(512), LDS_BYTES, stream, P);
    HIP_CHECK(hipGetLastError());
}
```

```cpp
#include <hip/hip_runtime.h>
#include <rccl/rccl.h>
#include <cstdio>
#include <cstdint>
#include <type_traits>

#define HIP_CHECK(expr) do { hipError_t _e = (expr); if (_e != hipSuccess) { \
    fprintf(stderr, "HIP error %s at %s:%d\n", hipGetErrorString(_e), __FILE__, __LINE__); \
    return; } } while (0)

#define SCH_RATIO8 24
#define SCH_LEAD8 72
#define SCH_XO8 420
#define SCH_OUT8 1000
#define SCH_OUTR8 8
#define SCH_XOC8 160
#define SCH_XOR8 4
#define SCH_XORC8 30

namespace {

typedef _Float16 h8 __attribute__((ext_vector_type(8)));
typedef float f4 __attribute__((ext_vector_type(4)));
typedef unsigned u4 __attribute__((ext_vector_type(4)));
#define LAS __attribute__((address_space(3)))

constexpr int NR = 8;
constexpr int NDIM = 8192;
constexpr int KSH = 1024;
#define NREL_V 2
constexpr int NREL = NREL_V;
#define NPUSH_WG 28
#define PUSH_LIM 6
constexpr int NPUSH = NPUSH_WG;
static_assert(NPUSH % 7 == 0 && NPUSH <= 32, "pusher count");
constexpr int NCH_IN = 128 + 256 + NREL * 32;
constexpr int NCH_XI = 128 + (8 - NREL) * 32 + NREL * 8;
constexpr int NCH_X = NCH_XI + 128;
#define NDIR_V 12
constexpr int NDIR = NDIR_V;
constexpr int GRID = 256;
constexpr int NGW = GRID - NPUSH;
constexpr int NCONV = 768;
constexpr unsigned long long DEADLINE_TICKS = 20000000ull;

constexpr unsigned OFF_TICKET = 0, OFF_TMO = 128, OFF_TQ = 256;
constexpr unsigned OFF_LKT = 1024;
constexpr unsigned OFF_LXF = 4096;
constexpr unsigned OFF_LWF = 8192;
constexpr unsigned OFF_LOF = 12288;
constexpr unsigned OFF_DONE = 16384;
constexpr unsigned OFF_XF = 65536;
constexpr unsigned OFF_WF = 65536 + 4096;
constexpr unsigned OFF_DBG = 262144;
constexpr unsigned MIB = 1u << 20;
constexpr unsigned OFF_XL = 1 * MIB;
constexpr unsigned OFF_WL = 17 * MIB;
constexpr unsigned OFF_XR = 33 * MIB;
constexpr unsigned OFF_WR = 65 * MIB;
constexpr unsigned PAR_STRIDE = 96 * MIB;
constexpr unsigned OFF_OS = 33 * MIB + 2 * PAR_STRIDE;
constexpr unsigned WS_USED = OFF_OS + 16 * MIB;

constexpr int STAGE_B = 49152;
constexpr int L_CTL = 3 * STAGE_B;
constexpr int L_Q = L_CTL;
constexpr int L_PF = L_CTL + 1024;
constexpr int L_RED = L_CTL + 1792;
constexpr int L_BC = L_CTL + 1856;
constexpr int L_META = L_CTL + 1920;
constexpr int LDS_BYTES = L_CTL + 2048;

#define OWN_AUX 17
#define SCH_XF0 -4
#define GLDS_AUX 16

struct ConvTab { unsigned v[NCONV]; };
constexpr int conv_key(int id) {
    const int b = id / 96, r = id % 96;
    int k = r < 32 ? (b ? b * SCH_RATIO8 - 4 : SCH_XF0) : ((r - 32) >> 1) * 8 + b * SCH_RATIO8 - (b >= 8 - NREL ? SCH_LEAD8 : 0);
    k += 64;
    if (r >= 32 && b == 0 && ((r - 32) >> 1) < NDIR) k = 0;
    return k < 0 ? 0 : (k > 2047 ? 2047 : k);
}
constexpr ConvTab make_conv_tab() {
    ConvTab t{}; int cnt[2049] = {};
    for (int i = 0; i < NCONV; ++i) cnt[conv_key(i) + 1]++;
    for (int k = 0; k < 2048; ++k) cnt[k + 1] += cnt[k];
    for (int i = 0; i < NCONV; ++i) t.v[cnt[conv_key(i)]++] = (unsigned)i;
    return t;
}
__constant__ const ConvTab conv_tab = make_conv_tab();

constexpr int chunk_key(int id, bool ing) {
    int k = 0;
    const int nW = ing ? 256 : (8 - NREL) * 32, r = id - 128, q = r - nW;
    if (!ing && id >= NCH_XI) k = SCH_OUT8 + (id - NCH_XI);
    else if (id < 128) { const int b = id >> 4, rbl = (id >> 1) & 7; k = rbl >= 4 ? (b ? b * SCH_RATIO8 - 4 : SCH_XF0) : (ing ? SCH_XO8 + (b * 8 + (id & 7)) * SCH_XOR8 / 8 : SCH_XOC8 + (b * 8 + (id & 7)) * SCH_XORC8 / 8); }
    else if (r < nW) k = (r & 31) * 8 + (r >> 5) * SCH_RATIO8;
    else if (ing) k = (q & 7) * 32 + 12 + (8 - NREL + (q >> 5)) * SCH_RATIO8;
    else k = (q & 7) * 32 + 12 + (8 - NREL + (q >> 3)) * SCH_RATIO8 - SCH_LEAD8;
    k += 64;
    return k < 0 ? 0 : (k > 2047 ? 2047 : k);
}
constexpr bool chunk_direct(int id) { return id >= 128 && id < 128 + NDIR; }
struct PushTab { unsigned v[2][512]; };
constexpr PushTab make_push_tab() {
    PushTab t{};
    for (int c = 0; c < 2; ++c) {
        const int n = c == 0 ? NCH_IN : NCH_X; int cnt[2049] = {};
        for (int i = 0; i < n; ++i) if (!chunk_direct(i)) cnt[chunk_key(i, c == 0) + 1]++;
        for (int k = 0; k < 2048; ++k) cnt[k + 1] += cnt[k];
        for (int i = 0; i < n; ++i) if (!chunk_direct(i)) t.v[c][cnt[chunk_key(i, c == 0)]++] = (unsigned)i;
    }
    return t;
}
__constant__ const PushTab push_tab = make_push_tab();

struct Params {
    const float* x; const float* w;
    void* ws_all[NR]; void* out_all[NR];
    int rank;
    int ratio8, lead8, xo8, xor8;
};

#define GAS __attribute__((address_space(1)))
__device__ __forceinline__ unsigned ld_sys(const unsigned* p) { return __hip_atomic_load((const GAS unsigned*)p, __ATOMIC_RELAXED, __HIP_MEMORY_SCOPE_SYSTEM); }
__device__ __forceinline__ unsigned ld_agent(const unsigned* p) { return __hip_atomic_load((const GAS unsigned*)p, __ATOMIC_RELAXED, __HIP_MEMORY_SCOPE_AGENT); }
__device__ __forceinline__ void st_sys(unsigned* p, unsigned v) { __hip_atomic_store((GAS unsigned*)p, v, __ATOMIC_RELAXED, __HIP_MEMORY_SCOPE_SYSTEM); }
__device__ __forceinline__ void st_agent(unsigned* p, unsigned v) { __hip_atomic_store((GAS unsigned*)p, v, __ATOMIC_RELAXED, __HIP_MEMORY_SCOPE_AGENT); }
__device__ __forceinline__ unsigned add_agent(unsigned* p) { return __hip_atomic_fetch_add((GAS unsigned*)p, 1u, __ATOMIC_RELAXED, __HIP_MEMORY_SCOPE_AGENT); }
__device__ __forceinline__ int rfl(int v) { return __builtin_amdgcn_readfirstlane(v); }
__device__ __forceinline__ unsigned rflu(unsigned v) { return (unsigned)__builtin_amdgcn_readfirstlane((int)v); }
__device__ __forceinline__ char* rfl_ptr(char* p) {
    const unsigned long long v = (unsigned long long)p;
    return (char*)(((unsigned long long)rflu((unsigned)(v >> 32)) << 32) | rflu((unsigned)v));
}
#define WAIT_V(n) asm volatile("s_waitcnt vmcnt(" #n ")" ::: "memory")
#define WAIT_L0() asm volatile("s_waitcnt lgkmcnt(0)" ::: "memory")
#define BAR() do { asm volatile("" ::: "memory"); __builtin_amdgcn_s_barrier(); asm volatile("" ::: "memory"); } while (0)

#define STAMP(ws, slot) do { } while (0)

__device__ __forceinline__ int rb32_of(int t, int rbl) {
    return rbl < 4 ? 4 * t + rbl : 4 * ((((t >> 2) ^ 1) << 2) + (rbl - 4)) + (t & 3);
}

__device__ __forceinline__ int block_scale(float am, LAS unsigned char* lds, int par, int& e) {
#pragma unroll
    for (int o = 32; o >= 1; o >>= 1) am = fmaxf(am, __shfl_xor(am, o));
    LAS float* red = (LAS float*)(lds + L_RED) + par * 8;
    if ((threadIdx.x & 63) == 0) red[threadIdx.x >> 6] = am;
    __syncthreads();
    float m = red[0];
#pragma unroll
    for (int i = 1; i < 8; ++i) m = fmaxf(m, red[i]);
    const int ex = __builtin_amdgcn_frexp_expf(m);
    e = rfl((15 - ex) & ~3);
    return rfl(m == 0.0f ? 0 : e + 257);
}
__device__ __forceinline__ float amax4(f4 v, float a) {
    return fmaxf(fmaxf(fmaxf(fabsf(v[0]), fabsf(v[1])), fmaxf(fabsf(v[2]), fabsf(v[3]))), a);
}
__device__ __forceinline__ _Float16 cvt1(float v, int e) { return (_Float16)__builtin_amdgcn_ldexpf(v, e); }

__device__ __forceinline__ float gelu_tanh(float y) {
    const float u = 0.7978845608028654f * (y + 0.044715f * y * y * y);
    const float ex = __expf(-2.0f * u);
    return y * __builtin_amdgcn_rcpf(1.0f + ex);
}

__global__ void __launch_bounds__(512, 2) fused_gemm_gelu(Params P) {
    extern __shared__ __attribute__((aligned(16))) unsigned char smem[];
    LAS unsigned char* lds = (LAS unsigned char*)smem;
    const int tid = threadIdx.x, lane = tid & 63, wid = rfl(tid >> 6);
    const int me = P.rank;
    char* ws = rfl_ptr((char*)P.ws_all[me]);
    unsigned* tmo = (unsigned*)(ws + OFF_TMO);

    const int bi = blockIdx.x;
    f4 v[16];
    auto load_item = [&](int it) {
        const int id = (int)conv_tab.v[it], b = id / 96, r = id % 96;
        if (r < 32) {
#pragma unroll
            for (int i = 0; i < 8; ++i) {
                const int q = i * 512 + tid, ks = q >> 11, row = (q >> 3) & 255, c = (q & 7) ^ (row & 7);
                const float* s = P.x + (size_t)(r * 256 + row) * KSH + b * 128 + ks * 64 + c * 8;
                v[2 * i] = *(const f4*)s; v[2 * i + 1] = *(const f4*)(s + 4);
            }
        } else {
            const int g2 = r - 32, cg = (g2 >> 1) + (g2 & 1) * 32;
#pragma unroll
            for (int j = 0; j < 8; ++j) v[j] = *(const f4*)(P.w + (size_t)(b * 128 + (tid >> 5) * 8 + j) * NDIM + cg * 128 + (tid & 31) * 4);
        }
    };
    if (bi >= NPUSH) load_item(bi - NPUSH);
    if (tid == 0) {
        const unsigned t = add_agent((unsigned*)(ws + OFF_TICKET));
        *(LAS unsigned*)(lds + L_BC) = t / GRID + 1u;
    }
    __syncthreads();
    const unsigned call = rflu(*(LAS unsigned*)(lds + L_BC));
    const unsigned seq = call & 0x3FFFFFu;
    const unsigned par = call & 1u;
    const unsigned xf_off = OFF_XF + par * 65536u, wf_off = OFF_WF + par * 65536u;
    const unsigned xr_off = OFF_XR + par * PAR_STRIDE, wr_off = OFF_WR + par * PAR_STRIDE;
    STAMP(ws, 0);

    if (bi < NPUSH) {
        bool dead = false;
        const unsigned loff = (unsigned)lane * 16u;
        u4 buf[4][4];
#pragma unroll
        for (int j = 0; j < 4; ++j)
#pragma unroll
            for (int i = 0; i < 4; ++i) buf[j][i] = (u4){0u, 0u, 0u, 0u};
        const int b7 = bi % 7, ci = b7 < 3 ? (bi / 7) * 3 + b7 : (bi / 7) * 4 + b7 - 3;
        if (wid < 4) {
        const int lx = b7 < 3 ? (ci * 4 + wid) % 3 : 3 + ((wid + ci) & 3);
        const int t = lx < 3 ? (me & 4) + ((me + 1 + lx) & 3) : ((me & 4) ^ 4) + ((me + lx) & 3);
        const bool ingrp = lx < 3;
        const int nch = (ingrp ? NCH_IN : NCH_X) - NDIR;
        unsigned* tkp = (unsigned*)(ws + OFF_LKT + lx * 128);
        const unsigned tkbase = (call - 1u) * (unsigned)(nch + 4 * NPUSH / 7);
        char* wsd = rfl_ptr((char*)P.ws_all[t & 7]);
        const unsigned* ord = push_tab.v[ingrp ? 0 : 1];
        char* outd = rfl_ptr((char*)P.out_all[t & 7]);
        struct Item { unsigned src, dst, wfl, dfl, out; };
        auto decode = [&](int tk) {
            const int id = (int)ord[tk];
            Item it; it.out = 0u;
            if (!ingrp && id >= NCH_XI) {
                const int q = id - NCH_XI, sg = q >> 4, c = q & 15;
                it.src = OFF_OS + (unsigned)((t & 3) * 128 + q) * 32768u; it.wfl = OFF_LOF + (unsigned)((t & 3) * 32 + sg * 4) * 4u;
                it.dst = (unsigned)(((me & 3) * 256 + c * 16) * NDIM + (me >> 2) * 4096 + sg * 512) * 4u;
                it.dfl = OFF_DONE + (unsigned)((me & 3) * 128 + q) * 4u; it.out = 1u;
            } else if (id < 128) {
                const int b = id >> 4, rbl = (id >> 1) & 7, ks = id & 1, rb = rb32_of(t, rbl);
                it.src = OFF_XL + (unsigned)(rb * 8 + b) * 65536u + ks * 32768u; it.wfl = OFF_LXF + (unsigned)(rb * 8 + b) * 4u;
                it.dst = xr_off + (unsigned)((rbl * 8 + me) * 8 + b) * 65536u + ks * 32768u;
                it.dfl = xf_off + (unsigned)((rbl * 2 + ks) * 64 + me * 8 + b) * 4u;
            } else {
                const int nW = ingrp ? 256 : (8 - NREL) * 32, r = id - 128, q = r - nW;
                if (r < nW || !ingrp) {
                    int b = r >> 5, g = r & 31;
                    if (r >= nW) { b = 8 - NREL + (q >> 3); g = 4 * (q & 7) + (((t & 3) - b) & 3); }
                    const int cg = (t >> 2) * 32 + g;
                    it.src = OFF_WL + (unsigned)(cg * 8 + b) * 32768u; it.wfl = OFF_LWF + (unsigned)(cg * 8 + b) * 4u;
                    it.dst = wr_off + (unsigned)((g * 8 + me) * 8 + b) * 32768u; it.dfl = wf_off + (unsigned)(g * 64 + me * 8 + b) * 4u;
                } else {
                    const int b = 8 - NREL + (q >> 5), ur = (((me >> 2) ^ 1) << 2) + ((q >> 3) & 3), g = 4 * (q & 7) + (((me & 3) - b) & 3);
                    it.src = it.dst = wr_off + (unsigned)((g * 8 + ur) * 8 + b) * 32768u;
                    it.wfl = it.dfl = wf_off + (unsigned)(g * 64 + ur * 8 + b) * 4u;
                }
            }
            return it;
        };
        auto draw = [&]() {
            unsigned v = 0;
            if (lane == 0) v = add_agent(tkp);
            return v;
        };
        int tk = (int)(rflu(draw()) - tkbase);
        Item cur{0u, 0u, 0u, 0u, 0u}, prv{0u, 0u, 0u, 0u, 0u}; unsigned cur_val = 0u, prv_val = 0u; bool cur_ok = false, prv_ok = false;
        unsigned fv = 0u;
        bool have = (unsigned)tk < (unsigned)nch;
        Item nxt{0u, 0u, 0u, 0u, 0u};
        if (have) nxt = decode(tk);
        for (;;) {
            cur_ok = false;
            if (have) {
                if (!prv_ok) {
                    const unsigned long long t0 = __builtin_amdgcn_s_memrealtime();
                    for (;;) {
                        const unsigned pv = ld_sys((const unsigned*)(ws + nxt.wfl + nxt.out * (unsigned)(lane & 3) * 4u));
                        fv = __all((pv >> 10) == seq) ? rflu(pv) : 0u;
                        if ((fv >> 10) == seq) break;
                        if (rflu(ld_agent(tmo)) == call || __builtin_amdgcn_s_memrealtime() - t0 > DEADLINE_TICKS) {
                            if (lane == 0) st_agent(tmo, call);
                            dead = true; break;
                        }
                        __builtin_amdgcn_s_sleep(2);
                    }
                    if (dead) break;
                }
                if ((fv >> 10) == seq) { cur = nxt; cur_val = fv; cur_ok = true; }
            } else if (!prv_ok) break;
            const __amdgpu_buffer_rsrc_t rl = __builtin_amdgcn_make_buffer_rsrc(ws, 0, cur_ok ? (int)WS_USED : 0, 0x00020000);
            const __amdgpu_buffer_rsrc_t rsp = __builtin_amdgcn_make_buffer_rsrc(prv.out ? outd : wsd, 0, prv_ok ? (prv.out ? 1024 * NDIM * 4 : (int)WS_USED) : 0, 0x00020000);
            const __amdgpu_buffer_rsrc_t rsc = __builtin_amdgcn_make_buffer_rsrc(cur.out ? outd : wsd, 0, cur_ok ? (cur.out ? 1024 * NDIM * 4 : (int)WS_USED) : 0, 0x00020000);
            const unsigned pks = prv.out ? NDIM * 4u : 2048u, cks = cur.out ? NDIM * 4u : 2048u;
            const unsigned pdo = prv.dst + loff + 8u * pks, cdo = cur.dst + loff;
            unsigned tkv = 0u, fvv = 0u;
            if (cur_ok && lane == 0) {
                const unsigned one = 1u;
                asm volatile("global_atomic_add %0, %1, %2, off sc0" : "=v"(tkv) : "v"(tkp), "v"(one) : "memory");
            }
#pragma unroll
            for (int st = 0; st < 8; ++st) {
                const int j = st & 3;
                if (st == 4) {
                    asm volatile("s_waitcnt vmcnt(32)" : "+v"(tkv) :: "memory");
                    if (cur_ok) { tk = (int)(rflu(tkv) - tkbase); have = (unsigned)tk < (unsigned)nch; if (have) nxt = decode(tk); }
                    fvv = ld_sys((const unsigned*)(ws + nxt.wfl + nxt.out * (unsigned)(lane & 3) * 4u));
                }
                if (st == 5) {
                    WAIT_V(10);
                    if (prv_ok && lane == 0) st_sys((unsigned*)(wsd + prv.dfl), prv_val);
                }
#pragma unroll
                for (int i = 0; i < 4; ++i) {
                    asm volatile("s_waitcnt vmcnt(%0)" :: "n"(PUSH_LIM) : "memory");
                    if (st < 4) __builtin_amdgcn_raw_buffer_store_b128(buf[j][i], rsp, (int)(pdo + (unsigned)(st * 2 + (i >> 1)) * pks + (unsigned)(i & 1) * 1024u), 0, 17);
                    else __builtin_amdgcn_raw_buffer_store_b128(buf[j][i], rsc, (int)(cdo + (unsigned)((st - 4) * 2 + (i >> 1)) * cks + (unsigned)(i & 1) * 1024u), 0, 17);
                    buf[j][i] = __builtin_amdgcn_raw_buffer_load_b128(rl, (int)(cur.src + loff + (unsigned)(st * 4096 + i * 1024)), 0, 16);
                }
            }
            fv = __all((fvv >> 10) == seq) ? rflu(fvv) : 0u;
            prv = cur; prv_val = cur_val; prv_ok = cur_ok;
        }
        }
        if (bi == 0 && wid == 4) {
            const unsigned* dp = (const unsigned*)(ws + OFF_DONE);
            const unsigned long long t0 = __builtin_amdgcn_s_memrealtime();
            bool bad = dead;
            while (!bad) {
                bool ok = true;
#pragma unroll
                for (int i = 0; i < 8; ++i) ok = ok && (ld_sys(dp + i * 64 + lane) >> 10) == seq;
                if (__all(ok)) break;
                if (rflu(ld_agent(tmo)) == call || __builtin_amdgcn_s_memrealtime() - t0 > DEADLINE_TICKS) {
                    if (lane == 0) st_agent(tmo, call);
                    bad = true; break;
                }
                __builtin_amdgcn_s_sleep(2);
            }
            __builtin_amdgcn_fence(__ATOMIC_ACQUIRE, "");
            if (bad && lane == 0) {
                float* o = (float*)P.out_all[me];
                o[1023 * (size_t)NDIM + (((me >> 2) ^ 1) * 4096 + 4095)] = __builtin_nanf("");
            }
        }
        return;
    }

    {
        const __amdgpu_buffer_rsrc_t rsl = __builtin_amdgcn_make_buffer_rsrc(ws, 0, (int)WS_USED, 0x00020000);
        int rp = 0;
        for (int it = bi - NPUSH; it < NCONV; it += NGW, rp ^= 1) {
            const int id = (int)conv_tab.v[it], b = id / 96, r = id % 96;
            int e, code; unsigned fo; float am = 0.f;
            bool direct = false; int dgrp = 0; unsigned dfo = 0u;
            if (r < 32) {
                const int rb = r;
#pragma unroll
                for (int i = 0; i < 16; ++i) am = amax4(v[i], am);
                code = block_scale(am, lds, rp, e);
                const unsigned base = OFF_XL + (unsigned)(rb * 8 + b) * 65536u + (unsigned)tid * 16u;
#pragma unroll
                for (int i = 0; i < 8; ++i) {
                    h8 o;
#pragma unroll
                    for (int j = 0; j < 4; ++j) { o[j] = cvt1(v[2 * i][j], e); o[4 + j] = cvt1(v[2 * i + 1][j], e); }
                    __builtin_amdgcn_raw_buffer_store_b128(__builtin_bit_cast(u4, o), rsl, (int)(base + i * 8192u), 0, 17);
                }
                fo = OFF_LXF + (unsigned)(rb * 8 + b) * 4u;
            } else {
                const int g2 = r - 32, cg = (g2 >> 1) + (g2 & 1) * 32;
                const int n4 = tid & 31, kc = tid >> 5;
#pragma unroll
                for (int j = 0; j < 8; ++j) am = amax4(v[j], am);
                code = block_scale(am, lds, rp, e);
#pragma unroll
                for (int e4 = 0; e4 < 4; ++e4) {
                    const int n = n4 * 4 + e4; h8 o;
#pragma unroll
                    for (int j = 0; j < 8; ++j) o[j] = cvt1(v[j][e4], e);
                    *(LAS h8*)(lds + (kc >> 3) * 16384 + n * 128 + (((kc & 7) ^ (n & 7)) << 4)) = o;
                }
                __syncthreads();
                const unsigned base = OFF_WL + (unsigned)(cg * 8 + b) * 32768u + (unsigned)tid * 16u;
                u4 d[4];
#pragma unroll
                for (int i = 0; i < 4; ++i) {
                    d[i] = *(LAS u4*)(lds + i * 8192 + tid * 16);
                    __builtin_amdgcn_raw_buffer_store_b128(d[i], rsl, (int)(base + i * 8192u), 0, 17);
                }
                fo = OFF_LWF + (unsigned)(cg * 8 + b) * 4u;
                if (b == 0 && (g2 >> 1) < NDIR) {
                    direct = true; dgrp = g2 & 1; dfo = wf_off + (unsigned)((g2 >> 1) * 64 + me * 8) * 4u;
                    const unsigned rbase = wr_off + (unsigned)(((g2 >> 1) * 8 + me) * 8) * 32768u + (unsigned)tid * 16u;
                    for (int q = 0; q < 4; ++q) {
                        const int t = dgrp * 4 + ((q + (g2 >> 1)) & 3);
                        if (t == me) continue;
                        const __amdgpu_buffer_rsrc_t rr = __builtin_amdgcn_make_buffer_rsrc(rfl_ptr((char*)P.ws_all[t & 7]), 0, (int)WS_USED, 0x00020000);
#pragma unroll
                        for (int i = 0; i < 4; ++i) __builtin_amdgcn_raw_buffer_store_b128(d[i], rr, (int)(rbase + i * 8192u), 0, 17);
                    }
                }
            }
            if (it + NGW < NCONV) {
                load_item(it + NGW);
                if (conv_tab.v[it + NGW] % 96 < 32) WAIT_V(16); else WAIT_V(8);
            } else WAIT_V(0);
            BAR();
            if (tid == 0) st_sys((unsigned*)(ws + fo), (seq << 10) | (unsigned)code);
            if (direct && tid == 0) {
                for (int q = 0; q < 4; ++q)
                    if (dgrp * 4 + q != me) st_sys((unsigned*)((char*)P.ws_all[(dgrp * 4 + q) & 7] + dfo), (seq << 10) | (unsigned)code);
            }
        }
    }
    STAMP(ws, 1);

    const int wm = wid >> 1, wn = wid & 1, fr = lane & 15, fq = lane >> 4;
    const int fo0 = fr * 128 + ((fq ^ (fr & 7)) << 4), fo1 = fo0 ^ 64;
    const int abase = wm * 8192, bbase = 32768 + wn * 8192;
    constexpr unsigned TQ_PER_CALL = 256u + (32u - NPUSH);
    int xcd = bi & 7;
    int u = (bi >> 3) - ((NPUSH - xcd + 7) >> 3);
    int ntile_done = 0;
    bool pool = u >= 28;
    for (;;) {
        if (pool) {
            if (tid == 0) {
                const unsigned o = add_agent((unsigned*)(ws + OFF_TQ));
                *(LAS unsigned*)(lds + L_BC + 4) = o - (call - 1u) * TQ_PER_CALL;
            }
            __syncthreads();
            const unsigned nx = rflu(*(LAS unsigned*)(lds + L_BC + 4));
            if (nx >= 32u) break;
            xcd = (int)(nx >> 2); u = 28 + (int)(nx & 3u);
        }
        pool = true;
        const int g = 4 * xcd + ((u & 15) >> 2);
        const int rbl = (u < 16 ? 4 : 0) + (u & 3);
        const int rb = rb32_of(me, rbl), cg = (me >> 2) * 32 + g;
        f4 acc[4][4];
#pragma unroll
        for (int m = 0; m < 4; ++m)
#pragma unroll
            for (int n = 0; n < 4; ++n) acc[m][n] = (f4){0.f, 0.f, 0.f, 0.f};
        const int ls = lane >> 3, lb = lane & 7;
        const unsigned* xfp = (const unsigned*)(ws + (ls == me ? OFF_LXF + (unsigned)(rb * 8 + lb) * 4u : xf_off + (unsigned)(rbl * 128 + lane) * 4u));
        const unsigned* xgp = (const unsigned*)(ws + (ls == me ? OFF_LXF + (unsigned)(rb * 8 + lb) * 4u : xf_off + (unsigned)(rbl * 128 + 64 + lane) * 4u));
        const unsigned* wfp = (const unsigned*)(ws + (ls == me ? OFF_LWF + (unsigned)(cg * 8 + lb) * 4u : wf_off + (unsigned)(g * 64 + lane) * 4u));
        const unsigned blk_a = ls == me ? OFF_XL + (unsigned)(rb * 8 + lb) * 65536u : xr_off + (unsigned)(rbl * 64 + lane) * 65536u;
        const unsigned blk_b = ls == me ? OFF_WL + (unsigned)(cg * 8 + lb) * 32768u : wr_off + (unsigned)(g * 64 + lane) * 32768u;
        unsigned long long seen = 0ull; int nq = 0, Ecur = 0, Emin = 0, haveE = 0, bad = 0;
        int polled = 0;
        int cs = 0, is = 0, avail = 0, ep = 0, allk = 0; unsigned qa = 0u, qb = 0u, meta = 0u, metav = 0u;
        int rd = 0, gst = 0;

        auto absorb = [&](unsigned xv, unsigned xw, unsigned wv) {
            unsigned long long rdy = __ballot((xv >> 10) == seq && (xw >> 10) == seq && (wv >> 10) == seq) & ~seen;
            seen |= rdy;
            if (rdy == 0ull) return;
            LAS unsigned* q = (LAS unsigned*)(lds + L_Q);
            const int cx = (int)(xv & 1023u), cw = (int)(wv & 1023u), el = cx + cw - 514;
            if (!haveE) {
                const unsigned long long nz = __ballot(cx != 0 && cw != 0) & rdy;
                if (nz) { haveE = 1; Ecur = __builtin_amdgcn_readlane(el, __builtin_ctzll(nz)); Emin = Ecur; }
            }
            const unsigned long long same = __ballot(cx != 0 && cw != 0 && el == Ecur) & rdy;
            if ((same >> lane) & 1ull) {
                const int pos = nq + __builtin_popcountll(same & ((1ull << lane) - 1ull));
                q[pos] = blk_a; q[64 + pos] = blk_b;
            }
            nq += __builtin_popcountll(same);
            rdy &= ~same;
            while (rdy) {
                const int l = __builtin_ctzll(rdy); rdy &= rdy - 1;
                if (__builtin_amdgcn_readlane(cx, l) == 0 || __builtin_amdgcn_readlane(cw, l) == 0) continue;
                const int e = __builtin_amdgcn_readlane(el, l);
                if (e > Emin + 60) continue;
                const int delta = e - Ecur; Ecur = e; Emin = min(Emin, e);
                if (lane == l) { q[nq] = blk_a | ((unsigned)delta & 0xFFFFu); q[64 + nq] = blk_b; }
                ++nq;
            }
        };
        auto publish = [&]() {
            const bool all = (seen == ~0ull) || bad;
            ((LAS unsigned*)(lds + L_META))[ep & 1] = (unsigned)nq | (all ? 128u : 0u) | (bad ? 256u : 0u) | ((unsigned)(Ecur + 1024) << 16);
        };
        auto fetch_req = [&]() {
            LAS unsigned* q = (LAS unsigned*)(lds + L_Q);
            qa = q[lane]; qb = q[64 + lane];
            metav = ((LAS unsigned*)(lds + L_META))[ep & 1];
            ++ep;
        };
        auto fetch_use = [&]() { meta = rflu(metav); avail = 2 * (int)(meta & 127u); allk = (int)((meta >> 7) & 1u); };
        const char* ga = nullptr; const char* gb = nullptr; int gdst = 0;
        const char* wsa = ws + wid * 4096 + lane * 16; const char* wsb = ws + wid * 2048 + lane * 16;
        auto issue_addr = [&]() {
            const unsigned ks = (unsigned)is & 1u;
            ga = wsa + (((unsigned)__builtin_amdgcn_readlane((int)qa, is >> 1) & 0xFFFF0000u) + ks * 32768u);
            gb = wsb + ((unsigned)__builtin_amdgcn_readlane((int)qb, is >> 1) + ks * 16384u);
            gdst = gst; gst = gst == 2 * STAGE_B ? 0 : gst + STAGE_B;
            ++is;
        };
#define GLDS(gp, lp, off) __builtin_amdgcn_global_load_lds((const unsigned*)(gp), (LAS unsigned*)(lp), 16, off, GLDS_AUX)
        auto issue_a = [&](int h) {
            if (h == 0) { GLDS(ga, lds + gdst + wid * 4096, 0); GLDS(ga, lds + gdst + wid * 4096, 1024); }
            else { GLDS(ga, lds + gdst + wid * 4096, 2048); GLDS(ga, lds + gdst + wid * 4096, 3072); }
        };
        auto issue_b = [&](int) { GLDS(gb, lds + gdst + 32768 + wid * 2048, 0); GLDS(gb, lds + gdst + 32768 + wid * 2048, 1024); };
        auto issue = [&]() { issue_addr(); issue_a(0); issue_a(1); issue_b(0); };
        h8 a0[4], b0[4], a1[4], b1[4];
        auto read_frags = [&](h8 (&af)[4], h8 (&bf)[4], int st, int fo) {
            LAS unsigned char* sp = lds + st + fo;
#pragma unroll
            for (int n = 0; n < 4; ++n) bf[n] = *(LAS h8*)(sp + bbase + n * 2048);
#pragma unroll
            for (int m = 0; m < 4; ++m) af[m] = *(LAS h8*)(sp + abase + m * 2048);
        };
#define MMA_ROW(m, af, bf) do { _Pragma("unroll") for (int n = 0; n < 4; ++n) \
            asm volatile("v_mfma_f32_16x16x32_f16 %0, %1, %2, %0" : "+v"(acc[m][n]) : "v"(bf[n]), "v"(af[m])); } while (0)
#define SCHED() __builtin_amdgcn_sched_barrier(0)
#define ACC_FENCE() do { SCHED(); asm volatile("s_nop 15\n\ts_nop 15" : "+v"(acc[0][0]), "+v"(acc[0][1]), "+v"(acc[0][2]), "+v"(acc[0][3]), \
            "+v"(acc[1][0]), "+v"(acc[1][1]), "+v"(acc[1][2]), "+v"(acc[1][3]), "+v"(acc[2][0]), "+v"(acc[2][1]), "+v"(acc[2][2]), "+v"(acc[2][3]), \
            "+v"(acc[3][0]), "+v"(acc[3][1]), "+v"(acc[3][2]), "+v"(acc[3][3])); SCHED(); } while (0)
        int pre = 0;
        auto step = [&]() {
            __builtin_amdgcn_s_waitcnt(0xC07F);
            MMA_ROW(0, a0, b0);
            SCHED();
            read_frags(a1, b1, rd, fo1);
            SCHED();
            MMA_ROW(1, a0, b0);
            SCHED();
            const int upd = allk ^ 1;
            if (upd && wid == 0) {
                if (polled == 2) { absorb(((LAS unsigned*)(lds + L_PF))[lane], ((LAS unsigned*)(lds + L_PF))[128 + lane], ((LAS unsigned*)(lds + L_PF))[64 + lane]); polled = 0; }
                publish();
            }
            SCHED();
            MMA_ROW(2, a0, b0);
            SCHED();
            pre = is > cs + 1 ? 1 : 0;
            const bool deep = is > cs + 2;
            const bool go = is < avail;
            if (go) issue_addr();
            rd = rd == 2 * STAGE_B ? 0 : rd + STAGE_B;
            SCHED();
            MMA_ROW(3, a0, b0);
            SCHED();
            if (deep) WAIT_V(6); else WAIT_V(0);
            __builtin_amdgcn_s_waitcnt(0xC07F);
            BAR();
            MMA_ROW(0, a1, b1);
            SCHED();
            if (upd) fetch_req();
            if (pre) read_frags(a0, b0, rd, fo0);
            SCHED();
            MMA_ROW(1, a1, b1);
            SCHED();
            if (upd && wid == 0) {
                if (polled == 1) polled = 2;
                else if (seen != ~0ull) {
                    __builtin_amdgcn_global_load_lds(xfp, (LAS unsigned*)(lds + L_PF), 4, 0, 17);
                    __builtin_amdgcn_global_load_lds(wfp, (LAS unsigned*)(lds + L_PF + 256), 4, 0, 17);
                    __builtin_amdgcn_global_load_lds(xgp, (LAS unsigned*)(lds + L_PF + 512), 4, 0, 17);
                    polled = 1;
                }
            }
            if (go) issue_a(0);
            SCHED();
            MMA_ROW(2, a1, b1);
            SCHED();
            if (go) issue_a(1);
            SCHED();
            MMA_ROW(3, a1, b1);
            SCHED();
            if (go) issue_b(0);
            if (upd) {
                fetch_use();
                while (is < avail && is <= cs + 3) issue();
            }
            ++cs;
        };
        auto fast_step = [&](auto first) {
            constexpr bool FIRST = decltype(first)::value;
            __builtin_amdgcn_s_waitcnt(0xC07F);
            MMA_ROW(0, a0, b0);
            SCHED();
            read_frags(a1, b1, rd, fo1);
            SCHED();
            MMA_ROW(1, a0, b0);
            SCHED();
            const int upd = allk ^ 1;
            if (FIRST && upd && wid == 0) {
                if (polled == 2) { absorb(((LAS unsigned*)(lds + L_PF))[lane], ((LAS unsigned*)(lds + L_PF))[128 + lane], ((LAS unsigned*)(lds + L_PF))[64 + lane]); polled = 0; }
                publish();
            }
            issue_addr();
            SCHED();
            MMA_ROW(2, a0, b0);
            rd = rd == 2 * STAGE_B ? 0 : rd + STAGE_B;
            MMA_ROW(3, a0, b0);
            SCHED();
            WAIT_V(6);
            __builtin_amdgcn_s_waitcnt(0xC07F);
            BAR();
            MMA_ROW(0, a1, b1);
            SCHED();
            read_frags(a0, b0, rd, fo0);
            if (FIRST && upd) {
                fetch_req();
                if (wid == 0) {
                    if (polled == 1) polled = 2;
                    else if (seen != ~0ull) {
                        __builtin_amdgcn_global_load_lds(xfp, (LAS unsigned*)(lds + L_PF), 4, 0, 17);
                        __builtin_amdgcn_global_load_lds(wfp, (LAS unsigned*)(lds + L_PF + 256), 4, 0, 17);
                        __builtin_amdgcn_global_load_lds(xgp, (LAS unsigned*)(lds + L_PF + 512), 4, 0, 17);
                        polled = 1;
                    }
                }
            }
            if (!FIRST) polled = polled == 1 ? 2 : polled;
            SCHED();
            MMA_ROW(1, a1, b1);
            SCHED();
            issue_a(0);
            SCHED();
            MMA_ROW(2, a1, b1);
            SCHED();
            issue_a(1);
            SCHED();
            MMA_ROW(3, a1, b1);
            SCHED();
            issue_b(0);
            if (FIRST && upd) fetch_use();
            ++cs;
        };
        int delta = 0;
        auto restart = [&]() {
            while (cs == is && !(allk && cs == avail)) {
                if (wid == 0) {
                    const int nq0 = nq;
                    WAIT_V(0); polled = 0;
                    const unsigned long long t0 = __builtin_amdgcn_s_memrealtime();
                    for (;;) {
                        const unsigned xv = ld_sys(xfp), xw = ld_sys(xgp), wv = ld_sys(wfp);
                        absorb(xv, xw, wv);
                        if (nq > nq0 || seen == ~0ull) break;
                        if (rflu(ld_agent(tmo)) == call || __builtin_amdgcn_s_memrealtime() - t0 > DEADLINE_TICKS) {
                            if (lane == 0) st_agent(tmo, call);
                            bad = 1; break;
                        }
                        __builtin_amdgcn_s_sleep(2);
                    }
                    publish();
                    WAIT_L0();
                }
                BAR();
                fetch_req(); fetch_use();
                while (is < avail && is <= cs + 2) issue();
            }
            pre = cs == is ? 2 : 1;
            if (pre == 1) {
                if (is > cs + 2) WAIT_V(12); else if (is > cs + 1) WAIT_V(6); else WAIT_V(0);
                BAR();
                read_frags(a0, b0, rd, fo0);
            }
        };
        auto prepare = [&]() {
            if (!pre) restart();
            delta = pre == 2 ? 0 : (int)(short)__builtin_amdgcn_readlane((int)qa, cs >> 1);
        };
        prepare();
        while (pre != 2) {
            const int dc = max(delta, -250), d1 = dc >> 1, d2 = dc - d1;
            const float s1 = __builtin_bit_cast(float, (unsigned)(127 + d1) << 23), s2 = __builtin_bit_cast(float, (unsigned)(127 + d2) << 23);
            ACC_FENCE();
#pragma unroll
            for (int m = 0; m < 4; ++m)
#pragma unroll
                for (int n = 0; n < 4; ++n) acc[m][n] = (acc[m][n] * s1) * s2;
            ACC_FENCE();
            do {
                delta = 0;
                while (is == cs + 3 && avail >= cs + 5 && delta == 0) {
                    fast_step(std::true_type{});
                    fast_step(std::false_type{});
                    delta = (int)(short)__builtin_amdgcn_readlane((int)qa, cs >> 1);
                }
                if (delta == 0) {
                    step();
                    if (!pre) restart();
                    step();
                    prepare();
                }
            } while (pre != 2 && delta == 0);
        }
        ACC_FENCE();
        STAMP(ws, 2 + 2 * ntile_done);
        {
            const int E = (int)(meta >> 16) - 1024;
            const bool isbad = (meta & 256u) != 0u || rflu(ld_agent(tmo)) == call;
            char* ob = rfl_ptr((char*)P.out_all[me]);
            const __amdgpu_buffer_rsrc_t ro = __builtin_amdgcn_make_buffer_rsrc(ob, 0, 1024 * NDIM * 4, 0x00020000);
            const __amdgpu_buffer_rsrc_t rw = __builtin_amdgcn_make_buffer_rsrc(ws, 0, (int)WS_USED, 0x00020000);
            const unsigned o0 = (unsigned)(((rbl & 3) * 256 + wm * 64 + fr) * NDIM + cg * 128 + wn * 64 + fq * 4) * 4u;
            const unsigned o1 = OFF_OS + (unsigned)((rbl & 3) * 8 + (g >> 2)) * 524288u + (unsigned)((wm * 64 + fr) * 512 + (g & 3) * 128 + wn * 64 + fq * 4) * 4u;
            const float poison = isbad ? __builtin_nanf("") : 0.0f;
#pragma unroll
            for (int m = 0; m < 4; ++m)
#pragma unroll
                for (int n = 0; n < 4; ++n)
#pragma unroll
                    for (int j = 0; j < 4; ++j) acc[m][n][j] = gelu_tanh(__builtin_amdgcn_ldexpf(acc[m][n][j], -E)) + poison;
            if (rbl < 4) {
#pragma unroll
                for (int m = 0; m < 4; ++m)
#pragma unroll
                    for (int n = 0; n < 4; ++n)
                        __builtin_amdgcn_raw_buffer_store_b128(__builtin_bit_cast(u4, acc[m][n]), ro, (int)(o0 + (unsigned)(m * 16 * NDIM * 4 + n * 64)), 0, OWN_AUX);
            } else {
#pragma unroll
                for (int m = 0; m < 4; ++m)
#pragma unroll
                    for (int n = 0; n < 4; ++n)
                        __builtin_amdgcn_raw_buffer_store_b128(__builtin_bit_cast(u4, acc[m][n]), rw, (int)(o1 + (unsigned)(m * 16 * 2048 + n * 64)), 0, 17);
            }
            if (rbl >= 4) {
                WAIT_V(0);
                __syncthreads();
                if (tid == 0) st_sys((unsigned*)(ws + OFF_LOF) + (rbl & 3) * 32 + g, seq << 10);
            }
        }
        STAMP(ws, 3 + 2 * ntile_done);
        ++ntile_done;
    }
}

}

extern "C" void kernel_launch(int rank, int nranks, ncclComm_t comm,
                              hipStream_t stream,
                              void* const* d_in, const int* in_sizes, int n_in,
                              void* d_out, int out_size,
                              const int* in_dtypes, int out_dtype,
                              void* const* d_out_all,
                              void* const* d_ws_all, size_t ws_bytes) {
    (void)comm; (void)in_sizes; (void)n_in; (void)d_out; (void)out_size; (void)in_dtypes; (void)out_dtype;
    if (nranks != NR || (unsigned)rank >= (unsigned)NR || ws_bytes < (size_t)WS_USED) {
        fprintf(stderr, "kernel_launch: unsupported configuration (nranks %d, ws %zu)\n", nranks, ws_bytes);
        return;
    }
    HIP_CHECK(hipFuncSetAttribute((const void*)fused_gemm_gelu, hipFuncAttributeMaxDynamicSharedMemorySize, LDS_BYTES));
    Params P;
    P.x = (const float*)d_in[0]; P.w = (const float*)d_in[1];
    for (int i = 0; i < NR; ++i) { P.ws_all[i] = d_ws_all[i]; P.out_all[i] = d_out_all[i]; }
    P.rank = rank;
    P.ratio8 = SCH_RATIO8; P.lead8 = SCH_LEAD8; P.xo8 = SCH_XO8; P.xor8 = SCH_XOR8;
    hipLaunchKernelGGL(fused_gemm_gelu, dim3(GRID), dim3(512), LDS_BYTES, stream, P);
    HIP_CHECK(hipGetLastError());
}
```

```cpp
#include <hip/hip_runtime.h>
#include <rccl/rccl.h>
#include <cstdio>
#include <cstdint>
#include <type_traits>

#define HIP_CHECK(expr) do { hipError_t _e = (expr); if (_e != hipSuccess) { \
    fprintf(stderr, "HIP error %s at %s:%d\n", hipGetErrorString(_e), __FILE__, __LINE__); \
    return; } } while (0)

namespace {

typedef _Float16 h8 __attribute__((ext_vector_type(8)));
typedef float f4 __attribute__((ext_vector_type(4)));
typedef unsigned u4 __attribute__((ext_vector_type(4)));
#define LAS __attribute__((address_space(3)))

constexpr int NR = 8;
constexpr int NDIM = 8192;
constexpr int KSH = 1024;
constexpr int NREL = 2;
constexpr int NPUSH = 28;
constexpr int PUSH_LIM = 6;
static_assert(NPUSH % 7 == 0 && NPUSH <= 32, "pusher count");
constexpr int SCH_RATIO8 = 24;
constexpr int SCH_LEAD8 = 72;
constexpr int SCH_XO8 = 420, SCH_XOR8 = 4;
constexpr int SCH_XOC8 = 160, SCH_XORC8 = 30;
constexpr int SCH_OUT8 = 1000;
constexpr int xf_key(int b) { return b * SCH_RATIO8 - 4 > 128 + 4 * b ? b * SCH_RATIO8 - 4 : 128 + 4 * b; }
constexpr int STG_N = 80, STG_T = 250;
constexpr int NCH_IN = 128 + 256 + NREL * 32;
constexpr int NCH_XI = 128 + (8 - NREL) * 32 + NREL * 8;
constexpr int NCH_X = NCH_XI + 128;
constexpr int NDIR = 16;
constexpr int GRID = 256;
constexpr int NGW = GRID - NPUSH;
constexpr int NCONV = 768;
constexpr unsigned long long DEADLINE_TICKS = 20000000ull;

constexpr unsigned OFF_TICKET = 0, OFF_TMO = 128, OFF_TQ = 256;
constexpr unsigned OFF_LKT = 1024;
constexpr unsigned OFF_LXF = 4096;
constexpr unsigned OFF_LWF = 8192;
constexpr unsigned OFF_LOF = 12288;
constexpr unsigned OFF_DONE = 16384;
constexpr unsigned OFF_XF = 65536;
constexpr unsigned OFF_WF = 65536 + 4096;
constexpr unsigned MIB = 1u << 20;
constexpr unsigned OFF_XL = 1 * MIB;
constexpr unsigned OFF_WL = 17 * MIB;
constexpr unsigned OFF_XR = 33 * MIB;
constexpr unsigned OFF_WR = 65 * MIB;
constexpr unsigned PAR_STRIDE = 96 * MIB;
constexpr unsigned OFF_OS = 33 * MIB + 2 * PAR_STRIDE;
constexpr unsigned WS_USED = OFF_OS + 16 * MIB;

constexpr int STAGE_B = 49152;
constexpr int L_CTL = 3 * STAGE_B;
constexpr int L_Q = L_CTL;
constexpr int L_PF = L_CTL + 1024;
constexpr int L_RED = L_CTL + 1792;
constexpr int L_BC = L_CTL + 1856;
constexpr int L_META = L_CTL + 1920;
constexpr int LDS_BYTES = L_CTL + 2048;

constexpr int AUX_WT = 17;
constexpr int AUX_L2 = 16;

struct ConvTab { unsigned v[NCONV]; };
constexpr int conv_key(int id) {
    const int b = id / 96, r = id % 96;
    int k = r < 32 ? xf_key(b) : ((r - 32) >> 1) * 8 + b * SCH_RATIO8 - (b >= 8 - NREL ? SCH_LEAD8 : 0);
    k += 64;
    if (r >= 32 && b == 0 && ((r - 32) >> 1) < NDIR) k = 0;
    return k < 0 ? 0 : (k > 2047 ? 2047 : k);
}
constexpr ConvTab make_conv_tab() {
    ConvTab t{}; int cnt[2049] = {};
    for (int i = 0; i < NCONV; ++i) cnt[conv_key(i) + 1]++;
    for (int k = 0; k < 2048; ++k) cnt[k + 1] += cnt[k];
    for (int i = 0; i < NCONV; ++i) t.v[cnt[conv_key(i)]++] = (unsigned)i;
    return t;
}
__constant__ const ConvTab conv_tab = make_conv_tab();

constexpr int chunk_key(int id, bool ing) {
    int k = 0;
    const int nW = ing ? 256 : (8 - NREL) * 32, r = id - 128, q = r - nW;
    if (!ing && id >= NCH_XI) k = SCH_OUT8 + (id - NCH_XI);
    else if (id < 128) { const int b = id >> 4, rbl = (id >> 1) & 7; k = rbl >= 4 ? xf_key(b) : (ing ? SCH_XO8 + (b * 8 + (id & 7)) * SCH_XOR8 / 8 : SCH_XOC8 + (b * 8 + (id & 7)) * SCH_XORC8 / 8); }
    else if (r < nW) k = (r & 31) * 8 + (r >> 5) * SCH_RATIO8;
    else if (ing) k = (q & 7) * 32 + 12 + (8 - NREL + (q >> 5)) * SCH_RATIO8;
    else k = (q & 7) * 32 + 12 + (8 - NREL + (q >> 3)) * SCH_RATIO8 - SCH_LEAD8;
    k += 64;
    return k < 0 ? 0 : (k > 2047 ? 2047 : k);
}
constexpr bool chunk_direct(int id) { return id >= 128 && id < 128 + NDIR; }
struct PushTab { unsigned v[2][512]; };
constexpr PushTab make_push_tab() {
    PushTab t{};
    for (int c = 0; c < 2; ++c) {
        const int n = c == 0 ? NCH_IN : NCH_X; int cnt[2049] = {};
        for (int i = 0; i < n; ++i) if (!chunk_direct(i)) cnt[chunk_key(i, c == 0) + 1]++;
        for (int k = 0; k < 2048; ++k) cnt[k + 1] += cnt[k];
        for (int i = 0; i < n; ++i) if (!chunk_direct(i)) t.v[c][cnt[chunk_key(i, c == 0)]++] = (unsigned)i;
    }
    return t;
}
__constant__ const PushTab push_tab = make_push_tab();

struct Params {
    const float* x; const float* w;
    void* ws_all[NR]; void* out_all[NR];
    int rank;
};

#define GAS __attribute__((address_space(1)))
__device__ __forceinline__ unsigned ld_sys(const unsigned* p) { return __hip_atomic_load((const GAS unsigned*)p, __ATOMIC_RELAXED, __HIP_MEMORY_SCOPE_SYSTEM); }
__device__ __forceinline__ unsigned ld_agent(const unsigned* p) { return __hip_atomic_load((const GAS unsigned*)p, __ATOMIC_RELAXED, __HIP_MEMORY_SCOPE_AGENT); }
__device__ __forceinline__ void st_sys(unsigned* p, unsigned v) { __hip_atomic_store((GAS unsigned*)p, v, __ATOMIC_RELAXED, __HIP_MEMORY_SCOPE_SYSTEM); }
__device__ __forceinline__ void st_agent(unsigned* p, unsigned v) { __hip_atomic_store((GAS unsigned*)p, v, __ATOMIC_RELAXED, __HIP_MEMORY_SCOPE_AGENT); }
__device__ __forceinline__ unsigned add_agent(unsigned* p) { return __hip_atomic_fetch_add((GAS unsigned*)p, 1u, __ATOMIC_RELAXED, __HIP_MEMORY_SCOPE_AGENT); }
__device__ __forceinline__ int rfl(int v) { return __builtin_amdgcn_readfirstlane(v); }
__device__ __forceinline__ unsigned rflu(unsigned v) { return (unsigned)__builtin_amdgcn_readfirstlane((int)v); }
__device__ __forceinline__ char* rfl_ptr(char* p) {
    const unsigned long long v = (unsigned long long)p;
    return (char*)(((unsigned long long)rflu((unsigned)(v >> 32)) << 32) | rflu((unsigned)v));
}
#define WAIT_V(n) asm volatile("s_waitcnt vmcnt(" #n ")" ::: "memory")
#define WAIT_L0() asm volatile("s_waitcnt lgkmcnt(0)" ::: "memory")
#define BAR() do { asm volatile("" ::: "memory"); __builtin_amdgcn_s_barrier(); asm volatile("" ::: "memory"); } while (0)


__device__ __forceinline__ int rb32_of(int t, int rbl) {
    return rbl < 4 ? 4 * t + rbl : 4 * ((((t >> 2) ^ 1) << 2) + (rbl - 4)) + (t & 3);
}

__device__ __forceinline__ int block_scale(unsigned am, LAS unsigned char* lds, int par, int& e) {
#pragma unroll
    for (int o = 32; o >= 1; o >>= 1) am = max(am, (unsigned)__shfl_xor((int)am, o));
    LAS unsigned* red = (LAS unsigned*)(lds + L_RED) + par * 8;
    if ((threadIdx.x & 63) == 0) red[threadIdx.x >> 6] = am;
    __syncthreads();
    unsigned m = red[0];
#pragma unroll
    for (int i = 1; i < 8; ++i) m = max(m, red[i]);
    const int ex = __builtin_amdgcn_frexp_expf(__builtin_bit_cast(float, m));
    e = rfl((15 - ex) & ~3);
    return rfl(m == 0u ? 0 : e + 257);
}
__device__ __forceinline__ unsigned amax4(f4 v, unsigned a) {
    const u4 b = __builtin_bit_cast(u4, v) & 0x7fffffffu;
    return max(max(max(b[0], b[1]), max(b[2], b[3])), a);
}
__device__ __forceinline__ _Float16 cvt1(float v, int e) { return (_Float16)__builtin_amdgcn_ldexpf(v, e); }

__device__ __forceinline__ float gelu_tanh(float y) {
    const float u = 0.7978845608028654f * (y + 0.044715f * y * y * y);
    const float ex = __expf(-2.0f * u);
    return y * __builtin_amdgcn_rcpf(1.0f + ex);
}

__global__ void __launch_bounds__(512, 2) fused_gemm_gelu(Params P) {
    extern __shared__ __attribute__((aligned(16))) unsigned char smem[];
    LAS unsigned char* lds = (LAS unsigned char*)smem;
    const int tid = threadIdx.x, lane = tid & 63, wid = rfl(tid >> 6);
    const int me = P.rank;
    char* ws = rfl_ptr((char*)P.ws_all[me]);
    unsigned* tmo = (unsigned*)(ws + OFF_TMO);

    const int bi = blockIdx.x;
    f4 v[16];
    auto load_item = [&](int it) {
        const int id = (int)conv_tab.v[it], b = id / 96, r = id % 96;
        if (r < 32) {
#pragma unroll
            for (int i = 0; i < 8; ++i) {
                const int q = i * 512 + tid, ks = q >> 11, row = (q >> 3) & 255, c = (q & 7) ^ (row & 7);
                const float* s = P.x + (size_t)(r * 256 + row) * KSH + b * 128 + ks * 64 + c * 8;
                v[2 * i] = *(const f4*)s; v[2 * i + 1] = *(const f4*)(s + 4);
            }
        } else {
            const int g2 = r - 32, cg = (g2 >> 1) + (g2 & 1) * 32;
#pragma unroll
            for (int j = 0; j < 8; ++j) v[j] = *(const f4*)(P.w + (size_t)(b * 128 + (tid >> 5) * 8 + j) * NDIM + cg * 128 + (tid & 31) * 4);
        }
    };
    if (bi - NPUSH >= STG_N) {
        const unsigned long long t0 = __builtin_amdgcn_s_memrealtime();
        while (__builtin_amdgcn_s_memrealtime() - t0 < (unsigned long long)(STG_T * ((bi - NPUSH) / STG_N))) __builtin_amdgcn_s_sleep(4);
    }
    if (bi >= NPUSH) load_item(bi - NPUSH);
    if (tid == 0) {
        const unsigned t = add_agent((unsigned*)(ws + OFF_TICKET));
        *(LAS unsigned*)(lds + L_BC) = t / GRID + 1u;
    }
    __syncthreads();
    const unsigned call = rflu(*(LAS unsigned*)(lds + L_BC));
    const unsigned seq = call % 0x3FFFFFu + 1u;
    const unsigned par = call & 1u;
    const unsigned xf_off = OFF_XF + par * 65536u, wf_off = OFF_WF + par * 65536u;
    const unsigned xr_off = OFF_XR + par * PAR_STRIDE, wr_off = OFF_WR + par * PAR_STRIDE;

    if (bi < NPUSH) {
        bool dead = false;
        const unsigned loff = (unsigned)lane * 16u;
        u4 buf[4][4];
#pragma unroll
        for (int j = 0; j < 4; ++j)
#pragma unroll
            for (int i = 0; i < 4; ++i) buf[j][i] = (u4){0u, 0u, 0u, 0u};
        const int b7 = bi % 7, ci = b7 < 3 ? (bi / 7) * 3 + b7 : (bi / 7) * 4 + b7 - 3;
        if (wid < 4) {
        const int lx = b7 < 3 ? (ci * 4 + wid) % 3 : 3 + ((wid + ci) & 3);
        const int t = lx < 3 ? (me & 4) + ((me + 1 + lx) & 3) : ((me & 4) ^ 4) + ((me + lx) & 3);
        const bool ingrp = lx < 3;
        const int nch = (ingrp ? NCH_IN : NCH_X) - NDIR;
        unsigned* tkp = (unsigned*)(ws + OFF_LKT + lx * 128);
        const unsigned tkbase = (call - 1u) * (unsigned)(nch + 4 * NPUSH / 7);
        char* wsd = rfl_ptr((char*)P.ws_all[t & 7]);
        const unsigned* ord = push_tab.v[ingrp ? 0 : 1];
        char* outd = rfl_ptr((char*)P.out_all[t & 7]);
        struct Item { unsigned src, dst, wfl, dfl, out; };
        auto decode = [&](int tk) {
            const int id = (int)ord[tk];
            Item it; it.out = 0u;
            if (!ingrp && id >= NCH_XI) {
                const int q = id - NCH_XI, sg = q >> 4, c = q & 15;
                it.src = OFF_OS + (unsigned)((t & 3) * 128 + q) * 32768u; it.wfl = OFF_LOF + (unsigned)((t & 3) * 32 + sg * 4) * 4u;
                it.dst = (unsigned)(((me & 3) * 256 + c * 16) * NDIM + (me >> 2) * 4096 + sg * 512) * 4u;
                it.dfl = OFF_DONE + (unsigned)((me & 3) * 128 + q) * 4u; it.out = 1u;
            } else if (id < 128) {
                const int b = id >> 4, rbl = (id >> 1) & 7, ks = id & 1, rb = rb32_of(t, rbl);
                it.src = OFF_XL + (unsigned)(rb * 8 + b) * 65536u + ks * 32768u; it.wfl = OFF_LXF + (unsigned)(rb * 8 + b) * 4u;
                it.dst = xr_off + (unsigned)((rbl * 8 + me) * 8 + b) * 65536u + ks * 32768u;
                it.dfl = xf_off + (unsigned)((rbl * 2 + ks) * 64 + me * 8 + b) * 4u;
            } else {
                const int nW = ingrp ? 256 : (8 - NREL) * 32, r = id - 128, q = r - nW;
                if (r < nW || !ingrp) {
                    int b = r >> 5, g = r & 31;
                    if (r >= nW) { b = 8 - NREL + (q >> 3); g = 4 * (q & 7) + (((t & 3) - b) & 3); }
                    const int cg = (t >> 2) * 32 + g;
                    it.src = OFF_WL + (unsigned)(cg * 8 + b) * 32768u; it.wfl = OFF_LWF + (unsigned)(cg * 8 + b) * 4u;
                    it.dst = wr_off + (unsigned)((g * 8 + me) * 8 + b) * 32768u; it.dfl = wf_off + (unsigned)(g * 64 + me * 8 + b) * 4u;
                } else {
                    const int b = 8 - NREL + (q >> 5), ur = (((me >> 2) ^ 1) << 2) + ((q >> 3) & 3), g = 4 * (q & 7) + (((me & 3) - b) & 3);
                    it.src = it.dst = wr_off + (unsigned)((g * 8 + ur) * 8 + b) * 32768u;
                    it.wfl = it.dfl = wf_off + (unsigned)(g * 64 + ur * 8 + b) * 4u;
                }
            }
            return it;
        };
        auto draw = [&]() {
            unsigned v = 0;
            if (lane == 0) v = add_agent(tkp);
            return v;
        };
        int tk = (int)(rflu(draw()) - tkbase);
        Item cur{0u, 0u, 0u, 0u, 0u}, prv{0u, 0u, 0u, 0u, 0u}; unsigned cur_val = 0u, prv_val = 0u; bool cur_ok = false, prv_ok = false;
        unsigned fv = 0u;
        bool have = (unsigned)tk < (unsigned)nch;
        Item nxt{0u, 0u, 0u, 0u, 0u};
        if (have) nxt = decode(tk);
        for (;;) {
            cur_ok = false;
            if (have) {
                if (!prv_ok) {
                    const unsigned long long t0 = __builtin_amdgcn_s_memrealtime();
                    for (;;) {
                        const unsigned pv = ld_sys((const unsigned*)(ws + nxt.wfl + nxt.out * (unsigned)(lane & 3) * 4u));
                        fv = __all((pv >> 10) == seq) ? rflu(pv) : 0u;
                        if ((fv >> 10) == seq) break;
                        if (rflu(ld_agent(tmo)) == call || __builtin_amdgcn_s_memrealtime() - t0 > DEADLINE_TICKS) {
                            if (lane == 0) st_agent(tmo, call);
                            dead = true; break;
                        }
                        __builtin_amdgcn_s_sleep(2);
                    }
                    if (dead) break;
                }
                if ((fv >> 10) == seq) { cur = nxt; cur_val = fv; cur_ok = true; }
            } else if (!prv_ok) break;
            const __amdgpu_buffer_rsrc_t rl = __builtin_amdgcn_make_buffer_rsrc(ws, 0, cur_ok ? (int)WS_USED : 0, 0x00020000);
            const __amdgpu_buffer_rsrc_t rsp = __builtin_amdgcn_make_buffer_rsrc(prv.out ? outd : wsd, 0, prv_ok ? (prv.out ? 1024 * NDIM * 4 : (int)WS_USED) : 0, 0x00020000);
            const __amdgpu_buffer_rsrc_t rsc = __builtin_amdgcn_make_buffer_rsrc(cur.out ? outd : wsd, 0, cur_ok ? (cur.out ? 1024 * NDIM * 4 : (int)WS_USED) : 0, 0x00020000);
            const unsigned pks = prv.out ? NDIM * 4u : 2048u, cks = cur.out ? NDIM * 4u : 2048u;
            const unsigned pdo = prv.dst + loff + 8u * pks, cdo = cur.dst + loff;
            unsigned tkv = 0u, fvv = 0u;
            if (cur_ok && lane == 0) {
                const unsigned one = 1u;
                asm volatile("global_atomic_add %0, %1, %2, off sc0" : "=v"(tkv) : "v"(tkp), "v"(one) : "memory");
            }
#pragma unroll
            for (int st = 0; st < 8; ++st) {
                const int j = st & 3;
                if (st == 4) {
                    asm volatile("s_waitcnt vmcnt(32)" : "+v"(tkv) :: "memory");
                    if (cur_ok) { tk = (int)(rflu(tkv) - tkbase); have = (unsigned)tk < (unsigned)nch; if (have) nxt = decode(tk); }
                    fvv = ld_sys((const unsigned*)(ws + nxt.wfl + nxt.out * (unsigned)(lane & 3) * 4u));
                }
                if (st == 5) {
                    WAIT_V(10);
                    if (prv_ok && lane == 0) st_sys((unsigned*)(wsd + prv.dfl), prv_val);
                }
#pragma unroll
                for (int i = 0; i < 4; ++i) {
                    asm volatile("s_waitcnt vmcnt(%0)" :: "n"(PUSH_LIM) : "memory");
                    if (st < 4) __builtin_amdgcn_raw_buffer_store_b128(buf[j][i], rsp, (int)(pdo + (unsigned)(st * 2 + (i >> 1)) * pks + (unsigned)(i & 1) * 1024u), 0, AUX_WT);
                    else __builtin_amdgcn_raw_buffer_store_b128(buf[j][i], rsc, (int)(cdo + (unsigned)((st - 4) * 2 + (i >> 1)) * cks + (unsigned)(i & 1) * 1024u), 0, AUX_WT);
                    buf[j][i] = __builtin_amdgcn_raw_buffer_load_b128(rl, (int)(cur.src + loff + (unsigned)(st * 4096 + i * 1024)), 0, AUX_L2);
                }
            }
            fv = __all((fvv >> 10) == seq) ? rflu(fvv) : 0u;
            prv = cur; prv_val = cur_val; prv_ok = cur_ok;
        }
        }
        if (bi == 0 && wid == 4) {
            const unsigned* dp = (const unsigned*)(ws + OFF_DONE);
            const unsigned long long t0 = __builtin_amdgcn_s_memrealtime();
            bool bad = dead;
            while (!bad) {
                bool ok = true;
#pragma unroll
                for (int i = 0; i < 8; ++i) ok = ok && (ld_sys(dp + i * 64 + lane) >> 10) == seq;
                if (__all(ok)) break;
                if (rflu(ld_agent(tmo)) == call || __builtin_amdgcn_s_memrealtime() - t0 > DEADLINE_TICKS) {
                    if (lane == 0) st_agent(tmo, call);
                    bad = true; break;
                }
                __builtin_amdgcn_s_sleep(2);
            }
            __builtin_amdgcn_fence(__ATOMIC_ACQUIRE, "");
            if (bad && lane == 0) {
                float* o = (float*)P.out_all[me];
                o[(me >> 2) * 4096] = __builtin_nanf("");
                o[1023 * (size_t)NDIM + (((me >> 2) ^ 1) * 4096 + 4095)] = __builtin_nanf("");
            }
        }
        return;
    }

    {
        const __amdgpu_buffer_rsrc_t rsl = __builtin_amdgcn_make_buffer_rsrc(ws, 0, (int)WS_USED, 0x00020000);
        int rp = 0;
        for (int it = bi - NPUSH; it < NCONV; it += NGW, rp ^= 1) {
            const int id = (int)conv_tab.v[it], b = id / 96, r = id % 96;
            int e, code; unsigned fo, am = 0u;
            bool direct = false; int dgrp = 0; unsigned dfo = 0u;
            if (r < 32) {
                const int rb = r;
#pragma unroll
                for (int i = 0; i < 16; ++i) am = amax4(v[i], am);
                code = block_scale(am, lds, rp, e);
                const unsigned base = OFF_XL + (unsigned)(rb * 8 + b) * 65536u + (unsigned)tid * 16u;
#pragma unroll
                for (int i = 0; i < 8; ++i) {
                    h8 o;
#pragma unroll
                    for (int j = 0; j < 4; ++j) { o[j] = cvt1(v[2 * i][j], e); o[4 + j] = cvt1(v[2 * i + 1][j], e); }
                    __builtin_amdgcn_raw_buffer_store_b128(__builtin_bit_cast(u4, o), rsl, (int)(base + i * 8192u), 0, AUX_WT);
                }
                fo = OFF_LXF + (unsigned)(rb * 8 + b) * 4u;
            } else {
                const int g2 = r - 32, cg = (g2 >> 1) + (g2 & 1) * 32;
                const int n4 = tid & 31, kc = tid >> 5;
#pragma unroll
                for (int j = 0; j < 8; ++j) am = amax4(v[j], am);
                code = block_scale(am, lds, rp, e);
#pragma unroll
                for (int e4 = 0; e4 < 4; ++e4) {
                    const int n = n4 * 4 + e4; h8 o;
#pragma unroll
                    for (int j = 0; j < 8; ++j) o[j] = cvt1(v[j][e4], e);
                    *(LAS h8*)(lds + (kc >> 3) * 16384 + n * 128 + (((kc & 7) ^ (n & 7)) << 4)) = o;
                }
                __syncthreads();
                const unsigned base = OFF_WL + (unsigned)(cg * 8 + b) * 32768u + (unsigned)tid * 16u;
                u4 d[4];
#pragma unroll
                for (int i = 0; i < 4; ++i) {
                    d[i] = *(LAS u4*)(lds + i * 8192 + tid * 16);
                    __builtin_amdgcn_raw_buffer_store_b128(d[i], rsl, (int)(base + i * 8192u), 0, AUX_WT);
                }
                fo = OFF_LWF + (unsigned)(cg * 8 + b) * 4u;
                if (b == 0 && (g2 >> 1) < NDIR) {
                    direct = true; dgrp = g2 & 1; dfo = wf_off + (unsigned)((g2 >> 1) * 64 + me * 8) * 4u;
                    const unsigned rbase = wr_off + (unsigned)(((g2 >> 1) * 8 + me) * 8) * 32768u + (unsigned)tid * 16u;
                    for (int q = 0; q < 4; ++q) {
                        const int t = dgrp * 4 + ((q + (g2 >> 1)) & 3);
                        if (t == me) continue;
                        const __amdgpu_buffer_rsrc_t rr = __builtin_amdgcn_make_buffer_rsrc(rfl_ptr((char*)P.ws_all[t & 7]), 0, (int)WS_USED, 0x00020000);
#pragma unroll
                        for (int i = 0; i < 4; ++i) __builtin_amdgcn_raw_buffer_store_b128(d[i], rr, (int)(rbase + i * 8192u), 0, AUX_WT);
                    }
                }
            }
            asm volatile("" ::: "memory");
            if (it + NGW < NCONV) {
                load_item(it + NGW);
                if (conv_tab.v[it + NGW] % 96 < 32) WAIT_V(16); else WAIT_V(8);
            } else WAIT_V(0);
            BAR();
            if (tid == 0) st_sys((unsigned*)(ws + fo), (seq << 10) | (unsigned)code);
            if (direct && tid == 0) {
                for (int q = 0; q < 4; ++q)
                    if (dgrp * 4 + q != me) st_sys((unsigned*)((char*)P.ws_all[(dgrp * 4 + q) & 7] + dfo), (seq << 10) | (unsigned)code);
            }
        }
    }

    const int wm = wid >> 1, wn = wid & 1, fr = lane & 15, fq = lane >> 4;
    const int fo0 = fr * 128 + ((fq ^ (fr & 7)) << 4), fo1 = fo0 ^ 64;
    const int abase = wm * 8192, bbase = 32768 + wn * 8192;
    constexpr unsigned TQ_PER_CALL = 256u + (32u - NPUSH);
    int xcd = bi & 7;
    int u = (bi >> 3) - ((NPUSH - xcd + 7) >> 3);
    bool pool = u >= 28;
    for (;;) {
        if (pool) {
            if (tid == 0) {
                const unsigned o = add_agent((unsigned*)(ws + OFF_TQ));
                *(LAS unsigned*)(lds + L_BC + 4) = o - (call - 1u) * TQ_PER_CALL;
            }
            __syncthreads();
            const unsigned nx = rflu(*(LAS unsigned*)(lds + L_BC + 4));
            if (nx >= 32u) break;
            xcd = (int)(nx >> 2); u = 28 + (int)(nx & 3u);
        }
        pool = true;
        const int g = 4 * xcd + ((u & 15) >> 2);
        const int rbl = (u < 16 ? 4 : 0) + (u & 3);
        const int rb = rb32_of(me, rbl), cg = (me >> 2) * 32 + g;
        f4 acc[4][4];
#pragma unroll
        for (int m = 0; m < 4; ++m)
#pragma unroll
            for (int n = 0; n < 4; ++n) acc[m][n] = (f4){0.f, 0.f, 0.f, 0.f};
        const int ls = lane >> 3, lb = lane & 7;
        const unsigned* xfp = (const unsigned*)(ws + (ls == me ? OFF_LXF + (unsigned)(rb * 8 + lb) * 4u : xf_off + (unsigned)(rbl * 128 + lane) * 4u));
        const unsigned* xgp = (const unsigned*)(ws + (ls == me ? OFF_LXF + (unsigned)(rb * 8 + lb) * 4u : xf_off + (unsigned)(rbl * 128 + 64 + lane) * 4u));
        const unsigned* wfp = (const unsigned*)(ws + (ls == me ? OFF_LWF + (unsigned)(cg * 8 + lb) * 4u : wf_off + (unsigned)(g * 64 + lane) * 4u));
        const unsigned blk_a = ls == me ? OFF_XL + (unsigned)(rb * 8 + lb) * 65536u : xr_off + (unsigned)(rbl * 64 + lane) * 65536u;
        const unsigned blk_b = ls == me ? OFF_WL + (unsigned)(cg * 8 + lb) * 32768u : wr_off + (unsigned)(g * 64 + lane) * 32768u;
        unsigned long long seen = 0ull; int nq = 0, Ecur = 0, Emin = 0, haveE = 0, bad = 0;
        int polled = 0;
        int cs = 0, is = 0, avail = 0, ep = 0, allk = 0; unsigned qa = 0u, qb = 0u, meta = 0u, metav = 0u;
        int rd = 0, gst = 0;

        auto absorb = [&](unsigned xv, unsigned xw, unsigned wv) {
            unsigned long long rdy = __ballot((xv >> 10) == seq && (xw >> 10) == seq && (wv >> 10) == seq) & ~seen;
            seen |= rdy;
            if (rdy == 0ull) return;
            LAS unsigned* q = (LAS unsigned*)(lds + L_Q);
            const int cx = (int)(xv & 1023u), cw = (int)(wv & 1023u), el = cx + cw - 514;
            if (!haveE) {
                const unsigned long long nz = __ballot(cx != 0 && cw != 0) & rdy;
                if (nz) { haveE = 1; Ecur = __builtin_amdgcn_readlane(el, __builtin_ctzll(nz)); Emin = Ecur; }
            }
            const unsigned long long same = __ballot(cx == 0 || cw == 0 || el == Ecur) & rdy;
            if ((same >> lane) & 1ull) {
                const int pos = nq + __builtin_popcountll(same & ((1ull << lane) - 1ull));
                q[pos] = blk_a; q[64 + pos] = blk_b;
            }
            nq += __builtin_popcountll(same);
            rdy &= ~same;
            while (rdy) {
                const int l = __builtin_ctzll(rdy); rdy &= rdy - 1;
                const int e = __builtin_amdgcn_readlane(el, l);
                if (e > Emin + 60) continue;
                const int delta = e - Ecur; Ecur = e; Emin = min(Emin, e);
                if (lane == l) { q[nq] = blk_a | ((unsigned)delta & 0xFFFFu); q[64 + nq] = blk_b; }
                ++nq;
            }
        };
        auto publish = [&]() {
            const bool all = (seen == ~0ull) || bad;
            ((LAS unsigned*)(lds + L_META))[ep & 1] = (unsigned)nq | (all ? 128u : 0u) | (bad ? 256u : 0u) | ((unsigned)(Ecur + 1024) << 16);
        };
        auto fetch_req = [&]() {
            LAS unsigned* q = (LAS unsigned*)(lds + L_Q);
            qa = q[lane]; qb = q[64 + lane];
            metav = ((LAS unsigned*)(lds + L_META))[ep & 1];
            ++ep;
        };
        auto fetch_use = [&]() { meta = rflu(metav); avail = 2 * (int)(meta & 127u); allk = (int)((meta >> 7) & 1u); };
        const char* ga = nullptr; const char* gb = nullptr; int gdst = 0;
        const char* wsa = ws + wid * 4096 + lane * 16; const char* wsb = ws + wid * 2048 + lane * 16;
        auto issue_addr = [&]() {
            const unsigned ks = (unsigned)is & 1u;
            ga = wsa + (((unsigned)__builtin_amdgcn_readlane((int)qa, is >> 1) & 0xFFFF0000u) + ks * 32768u);
            gb = wsb + ((unsigned)__builtin_amdgcn_readlane((int)qb, is >> 1) + ks * 16384u);
            gdst = gst; gst = gst == 2 * STAGE_B ? 0 : gst + STAGE_B;
            ++is;
        };
#define GLDS(gp, lp, off) __builtin_amdgcn_global_load_lds((const unsigned*)(gp), (LAS unsigned*)(lp), 16, off, AUX_L2)
        auto issue_a = [&](int h) {
            if (h == 0) { GLDS(ga, lds + gdst + wid * 4096, 0); GLDS(ga, lds + gdst + wid * 4096, 1024); }
            else { GLDS(ga, lds + gdst + wid * 4096, 2048); GLDS(ga, lds + gdst + wid * 4096, 3072); }
        };
        auto issue_b = [&](int) { GLDS(gb, lds + gdst + 32768 + wid * 2048, 0); GLDS(gb, lds + gdst + 32768 + wid * 2048, 1024); };
        auto issue = [&]() { issue_addr(); issue_a(0); issue_a(1); issue_b(0); };
        h8 a0[4], b0[4], a1[4], b1[4];
        auto read_frags = [&](h8 (&af)[4], h8 (&bf)[4], int st, int fo) {
            LAS unsigned char* sp = lds + st + fo;
#pragma unroll
            for (int n = 0; n < 4; ++n) bf[n] = *(LAS h8*)(sp + bbase + n * 2048);
#pragma unroll
            for (int m = 0; m < 4; ++m) af[m] = *(LAS h8*)(sp + abase + m * 2048);
        };
#define MMA_ROW(m, af, bf) do { _Pragma("unroll") for (int n = 0; n < 4; ++n) \
            asm volatile("v_mfma_f32_16x16x32_f16 %0, %1, %2, %0" : "+v"(acc[m][n]) : "v"(bf[n]), "v"(af[m])); } while (0)
#define SCHED() __builtin_amdgcn_sched_barrier(0)
#define ACC_FENCE() do { SCHED(); asm volatile("s_nop 15\n\ts_nop 15" : "+v"(acc[0][0]), "+v"(acc[0][1]), "+v"(acc[0][2]), "+v"(acc[0][3]), \
            "+v"(acc[1][0]), "+v"(acc[1][1]), "+v"(acc[1][2]), "+v"(acc[1][3]), "+v"(acc[2][0]), "+v"(acc[2][1]), "+v"(acc[2][2]), "+v"(acc[2][3]), \
            "+v"(acc[3][0]), "+v"(acc[3][1]), "+v"(acc[3][2]), "+v"(acc[3][3])); SCHED(); } while (0)
        int pre = 0;
        auto step = [&]() {
            __builtin_amdgcn_s_waitcnt(0xC07F);
            MMA_ROW(0, a0, b0);
            SCHED();
            read_frags(a1, b1, rd, fo1);
            SCHED();
            MMA_ROW(1, a0, b0);
            SCHED();
            const int upd = allk ^ 1;
            if (upd && wid == 0) {
                if (polled == 2) { absorb(((LAS unsigned*)(lds + L_PF))[lane], ((LAS unsigned*)(lds + L_PF))[128 + lane], ((LAS unsigned*)(lds + L_PF))[64 + lane]); polled = 0; }
                publish();
            }
            SCHED();
            MMA_ROW(2, a0, b0);
            SCHED();
            pre = is > cs + 1 ? 1 : 0;
            const bool deep = is > cs + 2;
            const bool go = is < avail;
            if (go) issue_addr();
            rd = rd == 2 * STAGE_B ? 0 : rd + STAGE_B;
            SCHED();
            MMA_ROW(3, a0, b0);
            SCHED();
            if (deep) WAIT_V(6); else WAIT_V(0);
            __builtin_amdgcn_s_waitcnt(0xC07F);
            BAR();
            MMA_ROW(0, a1, b1);
            SCHED();
            if (upd) fetch_req();
            if (pre) read_frags(a0, b0, rd, fo0);
            SCHED();
            MMA_ROW(1, a1, b1);
            SCHED();
            if (upd && wid == 0) {
                if (polled == 1) polled = 2;
                else if (seen != ~0ull) {
                    __builtin_amdgcn_global_load_lds(xfp, (LAS unsigned*)(lds + L_PF), 4, 0, AUX_WT);
                    __builtin_amdgcn_global_load_lds(wfp, (LAS unsigned*)(lds + L_PF + 256), 4, 0, AUX_WT);
                    __builtin_amdgcn_global_load_lds(xgp, (LAS unsigned*)(lds + L_PF + 512), 4, 0, AUX_WT);
                    polled = 1;
                }
            }
            if (go) issue_a(0);
            SCHED();
            MMA_ROW(2, a1, b1);
            SCHED();
            if (go) issue_a(1);
            SCHED();
            MMA_ROW(3, a1, b1);
            SCHED();
            if (go) issue_b(0);
            if (upd) {
                fetch_use();
                while (is < avail && is <= cs + 3) issue();
            }
            ++cs;
        };
        auto fast_step = [&](auto first) {
            constexpr bool FIRST = decltype(first)::value;
            __builtin_amdgcn_s_waitcnt(0xC07F);
            MMA_ROW(0, a0, b0);
            SCHED();
            read_frags(a1, b1, rd, fo1);
            SCHED();
            MMA_ROW(1, a0, b0);
            SCHED();
            const int upd = allk ^ 1;
            if (FIRST && upd && wid == 0) {
                if (polled == 2) { absorb(((LAS unsigned*)(lds + L_PF))[lane], ((LAS unsigned*)(lds + L_PF))[128 + lane], ((LAS unsigned*)(lds + L_PF))[64 + lane]); polled = 0; }
                publish();
            }
            issue_addr();
            SCHED();
            MMA_ROW(2, a0, b0);
            rd = rd == 2 * STAGE_B ? 0 : rd + STAGE_B;
            MMA_ROW(3, a0, b0);
            SCHED();
            WAIT_V(6);
            __builtin_amdgcn_s_waitcnt(0xC07F);
            BAR();
            MMA_ROW(0, a1, b1);
            SCHED();
            read_frags(a0, b0, rd, fo0);
            if (FIRST && upd) {
                fetch_req();
                if (wid == 0) {
                    if (polled == 1) polled = 2;
                    else if (seen != ~0ull) {
                        __builtin_amdgcn_global_load_lds(xfp, (LAS unsigned*)(lds + L_PF), 4, 0, AUX_WT);
                        __builtin_amdgcn_global_load_lds(wfp, (LAS unsigned*)(lds + L_PF + 256), 4, 0, AUX_WT);
                        __builtin_amdgcn_global_load_lds(xgp, (LAS unsigned*)(lds + L_PF + 512), 4, 0, AUX_WT);
                        polled = 1;
                    }
                }
            }
            if (!FIRST) polled = polled == 1 ? 2 : polled;
            SCHED();
            MMA_ROW(1, a1, b1);
            SCHED();
            issue_a(0);
            SCHED();
            MMA_ROW(2, a1, b1);
            SCHED();
            issue_a(1);
            SCHED();
            MMA_ROW(3, a1, b1);
            SCHED();
            issue_b(0);
            if (FIRST && upd) fetch_use();
            ++cs;
        };
        int delta = 0;
        auto restart = [&]() {
            while (cs == is && !(allk && cs == avail)) {
                if (wid == 0) {
                    const int nq0 = nq;
                    WAIT_V(0); polled = 0;
                    const unsigned long long t0 = __builtin_amdgcn_s_memrealtime();
                    for (;;) {
                        const unsigned xv = ld_sys(xfp), xw = ld_sys(xgp), wv = ld_sys(wfp);
                        absorb(xv, xw, wv);
                        if (nq > nq0 || seen == ~0ull) break;
                        if (rflu(ld_agent(tmo)) == call || __builtin_amdgcn_s_memrealtime() - t0 > DEADLINE_TICKS) {
                            if (lane == 0) st_agent(tmo, call);
                            bad = 1; break;
                        }
                        __builtin_amdgcn_s_sleep(2);
                    }
                    publish();
                    WAIT_L0();
                }
                BAR();
                fetch_req(); fetch_use();
                while (is < avail && is <= cs + 2) issue();
            }
            pre = cs == is ? 2 : 1;
            if (pre == 1) {
                if (is > cs + 2) WAIT_V(12); else if (is > cs + 1) WAIT_V(6); else WAIT_V(0);
                BAR();
                read_frags(a0, b0, rd, fo0);
            }
        };
        auto prepare = [&]() {
            if (!pre) restart();
            delta = pre == 2 ? 0 : (int)(short)__builtin_amdgcn_readlane((int)qa, cs >> 1);
        };
        prepare();
        while (pre != 2) {
            const int dc = max(delta, -250), d1 = dc >> 1, d2 = dc - d1;
            const float s1 = __builtin_bit_cast(float, (unsigned)(127 + d1) << 23), s2 = __builtin_bit_cast(float, (unsigned)(127 + d2) << 23);
            ACC_FENCE();
#pragma unroll
            for (int m = 0; m < 4; ++m)
#pragma unroll
                for (int n = 0; n < 4; ++n) acc[m][n] = (acc[m][n] * s1) * s2;
            ACC_FENCE();
            do {
                delta = 0;
                while (is == cs + 3 && avail >= cs + 5 && delta == 0) {
                    fast_step(std::true_type{});
                    fast_step(std::false_type{});
                    delta = (int)(short)__builtin_amdgcn_readlane((int)qa, cs >> 1);
                }
                if (delta == 0) {
                    step();
                    if (!pre) restart();
                    step();
                    prepare();
                }
            } while (pre != 2 && delta == 0);
        }
        ACC_FENCE();
        {
            const int E = (int)(meta >> 16) - 1024;
            const bool isbad = (meta & 256u) != 0u || rflu(ld_agent(tmo)) == call;
            char* ob = rfl_ptr((char*)P.out_all[me]);
            const __amdgpu_buffer_rsrc_t ro = __builtin_amdgcn_make_buffer_rsrc(ob, 0, 1024 * NDIM * 4, 0x00020000);
            const __amdgpu_buffer_rsrc_t rw = __builtin_amdgcn_make_buffer_rsrc(ws, 0, (int)WS_USED, 0x00020000);
            const unsigned o0 = (unsigned)(((rbl & 3) * 256 + wm * 64 + fr) * NDIM + cg * 128 + wn * 64 + fq * 4) * 4u;
            const unsigned o1 = OFF_OS + (unsigned)((rbl & 3) * 8 + (g >> 2)) * 524288u + (unsigned)((wm * 64 + fr) * 512 + (g & 3) * 128 + wn * 64 + fq * 4) * 4u;
            const float poison = isbad ? __builtin_nanf("") : 0.0f;
#pragma unroll
            for (int m = 0; m < 4; ++m)
#pragma unroll
                for (int n = 0; n < 4; ++n)
#pragma unroll
                    for (int j = 0; j < 4; ++j) acc[m][n][j] = gelu_tanh(__builtin_amdgcn_ldexpf(acc[m][n][j], -E)) + poison;
            if (rbl < 4) {
#pragma unroll
                for (int m = 0; m < 4; ++m)
#pragma unroll
                    for (int n = 0; n < 4; ++n)
                        __builtin_amdgcn_raw_buffer_store_b128(__builtin_bit_cast(u4, acc[m][n]), ro, (int)(o0 + (unsigned)(m * 16 * NDIM * 4 + n * 64)), 0, AUX_WT);
            } else {
#pragma unroll
                for (int m = 0; m < 4; ++m)
#pragma unroll
                    for (int n = 0; n < 4; ++n)
                        __builtin_amdgcn_raw_buffer_store_b128(__builtin_bit_cast(u4, acc[m][n]), rw, (int)(o1 + (unsigned)(m * 16 * 2048 + n * 64)), 0, AUX_WT);
            }
            if (rbl >= 4) {
                WAIT_V(0);
                __syncthreads();
                if (tid == 0) st_sys((unsigned*)(ws + OFF_LOF) + (rbl & 3) * 32 + g, seq << 10);
            }
        }
    }
}

}


extern "C" void kernel_launch(int rank, int nranks, ncclComm_t comm,
                              hipStream_t stream,
                              void* const* d_in, const int* in_sizes, int n_in,
                              void* d_out, int out_size,
                              const int* in_dtypes, int out_dtype,
                              void* const* d_out_all,
                              void* const* d_ws_all, size_t ws_bytes) {
    (void)comm; (void)in_sizes; (void)n_in; (void)d_out; (void)out_size; (void)in_dtypes; (void)out_dtype;
    if (nranks != NR || (unsigned)rank >= (unsigned)NR || ws_bytes < (size_t)WS_USED) {
        fprintf(stderr, "kernel_launch: unsupported configuration (nranks %d, ws %zu)\n", nranks, ws_bytes);
        return;
    }
    HIP_CHECK(hipFuncSetAttribute((const void*)fused_gemm_gelu, hipFuncAttributeMaxDynamicSharedMemorySize, LDS_BYTES));
    Params P;
    P.x = (const float*)d_in[0]; P.w = (const float*)d_in[1];
    for (int i = 0; i < NR; ++i) { P.ws_all[i] = d_ws_all[i]; P.out_all[i] = d_out_all[i]; }
    P.rank = rank;
    hipLaunchKernelGGL(fused_gemm_gelu, dim3(GRID), dim3(512), LDS_BYTES, stream, P);
    HIP_CHECK(hipGetLastError());
}
```
